```python
import numpy as np

kernel_name = "gpu_ag_gemm_m4096_k4096_n16384_e4m3e4m3_dequant"


ABSMAX_RTOL = 0.03

COMPUTE_DTYPE = "float32"
EPILOGUE = "dequant"
IN_QUANT = None
OUT_QUANT = None
IN_FP8 = "e4m3e4m3"
IN_DTYPES = {"x": "e4m3", "w_mat": "e4m3",
             "scale_x": "float32", "scale_w": "float32"}
WORLD_SIZE = 8

def _snap_fp8(x, mant_bits, max_abs, min_normal):
    a = np.abs(x.astype(np.float64))
    _m, _e = np.frexp(a)
    q = np.where(a >= min_normal, np.ldexp(1.0, _e - 1 - mant_bits),
                 min_normal * 2.0 ** -mant_bits)
    snapped = np.minimum(np.round(a / q) * q, max_abs)
    return (np.sign(x) * snapped).astype(np.float32)

def setup_inputs():

    x = _snap_fp8(np.random.randn(4096, 4096).astype(np.float32), 3, 448.0, 2.0 ** -6)
    w_mat = _snap_fp8(np.random.randn(4096, 16384).astype(np.float32), 3, 448.0, 2.0 ** -6)
    scale_x = (0.02 * (1.0 + np.random.rand(1))).astype(np.float32)
    scale_w = (0.02 * (1.0 + np.random.rand(1))).astype(np.float32)
    return {"x": x, "w_mat": w_mat, "scale_x": scale_x, "scale_w": scale_w}

def reference(x, w_mat, scale_x, scale_w):
    acc = (x @ w_mat).astype(np.float32)
    y = (acc * (scale_x * scale_w)).astype(np.float32)
    return y
```

```cpp
#include <hip/hip_runtime.h>
#include <rccl/rccl.h>
#include <cstdio>

#define PF_N 0
#define LOOPV 1
#define ST_MODE 2
#define PF_P 3

namespace agg {
#define LAS __attribute__((address_space(3)))
#define GAS __attribute__((address_space(1)))
typedef int v8i __attribute__((ext_vector_type(8)));
typedef int v4i __attribute__((ext_vector_type(4)));
typedef int v2i __attribute__((ext_vector_type(2)));
typedef unsigned v4u __attribute__((ext_vector_type(4)));
typedef float v4f __attribute__((ext_vector_type(4)));

constexpr int NR = 8;
constexpr int MSH = 512, KD = 4096, NL = 2048;
constexpr int BM = 256, BN = 128, BK = 128, NT = KD / BK;
constexpr int A_ST = BM * BK, B_ST = BK * BN, ST = A_ST + B_ST, NSTAGE = 3;
constexpr int LDS_PF = NSTAGE * ST;
constexpr int LDS_CTL = LDS_PF + 256;
constexpr int LDS_BYTES = LDS_CTL + 64;
constexpr size_t OFF_CNT = 0;
constexpr size_t OFF_TMO = 32768;
constexpr size_t OFF_FLAG = 65536;
constexpr size_t OFF_STG = 1u << 20;
constexpr size_t STG_BYTES = (size_t)MSH * KD;
constexpr size_t OFF_DIAG = 8u << 20;
constexpr size_t WS_NEED = 16u << 20;
constexpr unsigned long long DEADLINE_TICKS = 50000000ull;

struct Params {
    const unsigned char* x; const unsigned char* w; const float* sx; const float* sw; float* out;
    char* ws[NR];
    int rank; int pad;
};

__device__ __forceinline__ unsigned long long rt() { return __builtin_amdgcn_s_memrealtime(); }
template <int N> __device__ __forceinline__ void waitv() { asm volatile("s_waitcnt vmcnt(%0)" :: "n"(N) : "memory"); }
__device__ __forceinline__ void dma16(unsigned ldsaddr, unsigned voff, const void* base) {
    asm volatile("s_mov_b32 m0, %0\n\ts_nop 0\n\tglobal_load_lds_dwordx4 %1, %2" :: "s"(ldsaddr), "v"(voff), "s"(base) : "memory");
}
__device__ __forceinline__ void dma4(unsigned ldsaddr, unsigned voff, const void* base) {
    asm volatile("s_mov_b32 m0, %0\n\ts_nop 0\n\tglobal_load_lds_dword %1, %2" :: "s"(ldsaddr), "v"(voff), "s"(base) : "memory");
}
__device__ __forceinline__ void bar() { asm volatile("" ::: "memory"); __builtin_amdgcn_s_barrier(); asm volatile("" ::: "memory"); }

#define STAMP(i) do {} while (0)

__global__ __launch_bounds__(512, 2) void ag_gemm(Params p) {
    extern __shared__ __attribute__((aligned(16))) char smem[];
    LAS char* l = (LAS char*)smem;
    const unsigned lbase = (unsigned)(size_t)l;
    const int tid = threadIdx.x, lane = tid & 63;
    const int wid = __builtin_amdgcn_readfirstlane(tid >> 6);
    const int b = blockIdx.x, g = b & 7, ti = b >> 3, rtile = ti & 1, ct = ti >> 1;
    const int src = (p.rank + g) & 7;
    STAMP(0);

    unsigned voffB[2];
#pragma unroll
    for (int j = 0; j < 2; ++j) {
        const int pb = wid * 2 + j, k = pb * 8 + (lane >> 3);
        const int ch = (lane & 7) ^ (((lane >> 4) & 3) | (((pb >> 1) & 1) << 2));
        voffB[j] = (unsigned)(k * NL + ct * BN + ch * 16);
    }
    auto issueB = [&](int kt, int slot) {
        const unsigned char* base = p.w + (size_t)kt * BK * NL;
#pragma unroll
        for (int j = 0; j < 2; ++j) dma16(lbase + slot * ST + A_ST + (wid * 2 + j) * 1024, voffB[j], base);
    };

    unsigned call;
    {
        const int kt = b >> 3, row = (b & 7) * 64 + (tid >> 3), ch = tid & 7;
        v4u v = *(const GAS v4u*)(p.x + (size_t)row * KD + kt * BK + ch * 16);
        if (tid == 0) {
            unsigned t = __hip_atomic_fetch_add((GAS unsigned*)(p.ws[p.rank] + OFF_CNT + b * 128), 1u, __ATOMIC_RELAXED, __HIP_MEMORY_SCOPE_AGENT);
            *(LAS unsigned*)(l + LDS_CTL) = t + 1;
        }
        issueB(0, 0); issueB(1, 1);
        asm volatile("s_waitcnt lgkmcnt(0)" ::: "memory");
        bar();
        call = __builtin_amdgcn_readfirstlane(*(LAS unsigned*)(l + LDS_CTL));
        char* dst = p.ws[p.rank] + OFF_STG + (call & 1) * STG_BYTES + (size_t)kt * (MSH * BK) + row * BK + ch * 16;
        asm volatile("global_store_dwordx4 %0, %1, off sc0 sc1\n\ts_nop 1" :: "v"(dst), "v"(v) : "memory");
        waitv<0>();
        bar();
        if (tid < NR && tid != p.rank)
            __hip_atomic_store((GAS unsigned*)(p.ws[tid] + OFF_FLAG + p.rank * 1024 + b * 4), call, __ATOMIC_RELAXED, __HIP_MEMORY_SCOPE_SYSTEM);
    }
    STAMP(1);

    const unsigned char* Abase; unsigned RS, KS;
    if (g == 0) { Abase = p.x; RS = KD; KS = BK; }
    else { Abase = (const unsigned char*)(p.ws[src] + OFF_STG + (call & 1) * STG_BYTES); RS = BK; KS = MSH * BK; }
    bool ok = true;
    if (g != 0) {
        if (wid == 0) {
            GAS unsigned* tmo = (GAS unsigned*)(p.ws[p.rank] + OFF_TMO);
            const char* fp = p.ws[p.rank] + OFF_FLAG + src * 1024 + lane * 16;
            const unsigned long long t0 = rt();
            unsigned good = 1;
            for (;;) {
                v4u f;
                asm volatile("global_load_dwordx4 %0, %1, off sc0 sc1\n\ts_waitcnt vmcnt(0)" : "=&v"(f) : "v"(fp) : "memory");
                const bool all = f.x >= call && f.y >= call && f.z >= call && f.w >= call;
                if (__all(all)) break;
                if (__builtin_amdgcn_readfirstlane(__hip_atomic_load(tmo, __ATOMIC_RELAXED, __HIP_MEMORY_SCOPE_AGENT)) == call || rt() - t0 > DEADLINE_TICKS) {
                    if (lane == 0) __hip_atomic_store(tmo, call, __ATOMIC_RELAXED, __HIP_MEMORY_SCOPE_AGENT);
                    good = 0; break;
                }
                __builtin_amdgcn_s_sleep(2);
            }
            __builtin_amdgcn_fence(__ATOMIC_ACQUIRE, "");
            asm volatile("s_waitcnt vmcnt(0)" ::: "memory");
            if (good && __builtin_amdgcn_readfirstlane(__hip_atomic_load(tmo, __ATOMIC_RELAXED, __HIP_MEMORY_SCOPE_AGENT)) == call) good = 0;
            if (lane == 0) *(LAS unsigned*)(l + LDS_CTL + 4) = good;
            asm volatile("s_waitcnt lgkmcnt(0)" ::: "memory");
        }
        bar();
        ok = __builtin_amdgcn_readfirstlane(*(LAS unsigned*)(l + LDS_CTL + 4)) != 0;
    }
    STAMP(2);

    const int wm = wid >> 1, wn = wid & 1;
    const int orow = src * MSH + rtile * BM + wm * 64 + (lane & 15), ocol = ct * BN + wn * 64 + 4 * (lane >> 4);
    float* optr = p.out + (size_t)orow * NL + ocol;
    if (!ok) {
        const float qn = __builtin_nanf("");
#pragma unroll
        for (int mi = 0; mi < 4; ++mi)
#pragma unroll
            for (int ni = 0; ni < 4; ++ni) *(v4f*)(optr + (size_t)mi * 16 * NL + ni * 16) = (v4f){qn, qn, qn, qn};
        waitv<0>();
        return;
    }

    unsigned voffA[4];
#pragma unroll
    for (int j = 0; j < 4; ++j) {
        const int pa = wid * 4 + j, row = pa * 8 + (lane >> 3);
        const int ch = (lane & 7) ^ ((pa & 1) * 4 + (lane >> 4));
        voffA[j] = (unsigned)(rtile * BM + row) * RS + ch * 16;
    }
    auto issueA = [&](int kt, int slot) {
        const unsigned char* base = Abase + (size_t)kt * KS;
#pragma unroll
        for (int j = 0; j < 4; ++j) dma16(lbase + slot * ST + (wid * 4 + j) * 1024, voffA[j], base);
    };
    auto duty = [&](int) -> int { return 0; };
    auto issueP = [&](int) {};

    issueA(0, 0); issueA(1, 1);

    const unsigned sA = (lane & 15) >> 1, g4 = lane >> 4;
    const unsigned aA0 = (unsigned)(wm * 64 + (lane & 15)) * 128 + ((g4 ^ sA) * 16), aA1 = aA0 ^ 64;
    const unsigned rB = (lane & 15) >> 1, sB = ((rB >> 1) & 3) | ((g4 & 1) << 2);
    unsigned aB[4];
#pragma unroll
    for (int ni = 0; ni < 4; ++ni) aB[ni] = A_ST + (16 * g4 + rB) * 128 + (((wn * 4 + ni) ^ sB) * 16) + (lane & 1) * 8;

    v4f acc[4][4];
#pragma unroll
    for (int mi = 0; mi < 4; ++mi)
#pragma unroll
        for (int ni = 0; ni < 4; ++ni) acc[mi][ni] = (v4f){0.f, 0.f, 0.f, 0.f};

    waitv<4>();
    bar();
    STAMP(3);
    int slot = 0, dprev = duty(-1);
    for (int k = 0; k < NT; ++k) {
        const unsigned so = slot * ST;
        v8i Af[4], Bf[4];
#pragma unroll
        for (int ni = 0; ni < 4; ++ni) {
            LAS char* q = l + so + aB[ni];
            const v2i b0 = __builtin_amdgcn_ds_read_tr8_b64_v2i32((LAS v2i*)(q));
            const v2i b1 = __builtin_amdgcn_ds_read_tr8_b64_v2i32((LAS v2i*)(q + 1024));
            const v2i b2 = __builtin_amdgcn_ds_read_tr8_b64_v2i32((LAS v2i*)(q + 8192));
            const v2i b3 = __builtin_amdgcn_ds_read_tr8_b64_v2i32((LAS v2i*)(q + 9216));
            Bf[ni] = (v8i){b0.x, b0.y, b1.x, b1.y, b2.x, b2.y, b3.x, b3.y};
        }
#pragma unroll
        for (int mi = 0; mi < 4; ++mi) {
            const v4i a0 = *(LAS v4i*)(l + so + aA0 + mi * 2048);
            const v4i a1 = *(LAS v4i*)(l + so + aA1 + mi * 2048);
            Af[mi] = (v8i){a0.x, a0.y, a0.z, a0.w, a1.x, a1.y, a1.z, a1.w};
        }
        const int s2 = slot == 0 ? 2 : slot - 1;
        int x = 0;
        if (k + 2 < NT) {
            issueA(k + 2, s2); issueB(k + 2, s2);
            const int dcur = duty(k);
            if (dcur) issueP(k);
            x = dprev + dcur;
            dprev = dcur;
        }
#pragma unroll
        for (int mi = 0; mi < 4; ++mi)
#pragma unroll
            for (int ni = 0; ni < 4; ++ni)
                acc[mi][ni] = __builtin_amdgcn_mfma_scale_f32_16x16x128_f8f6f4(Bf[ni], Af[mi], acc[mi][ni], 0, 0, 0, 0, 0, 0);
        asm volatile("s_waitcnt lgkmcnt(0)" ::: "memory");
        if (k + 2 < NT) {
            if (x == 0) waitv<6>(); else if (x == 1) waitv<7>(); else waitv<8>();
        } else waitv<0>();
        bar();
        slot = slot == 2 ? 0 : slot + 1;
    }
    STAMP(8);
    const float sc = p.sx[0] * p.sw[0];
#pragma unroll
    for (int mi = 0; mi < 4; ++mi)
#pragma unroll
        for (int ni = 0; ni < 4; ++ni) {
            const v4f v = acc[mi][ni] * sc;
            float* q = optr + (size_t)mi * 16 * NL + ni * 16;
            asm volatile("global_store_dwordx4 %0, %1, off sc1\n\ts_nop 1" :: "v"(q), "v"(v) : "memory");
        }
}
}

extern "C" void kernel_launch(int rank, int nranks, ncclComm_t comm, hipStream_t stream,
                              void* const* d_in, const int* in_sizes, int n_in,
                              void* d_out, int out_size, const int* in_dtypes, int out_dtype,
                              void* const* d_out_all, void* const* d_ws_all, size_t ws_bytes) {
    using namespace agg;
    (void)comm; (void)in_dtypes; (void)out_dtype; (void)d_out_all;
    if (nranks != NR || n_in != 4 || (unsigned)rank >= (unsigned)NR || ws_bytes < WS_NEED ||
        in_sizes[0] != MSH * KD || in_sizes[1] != KD * NL || out_size != NR * MSH * NL) {
        fprintf(stderr, "ag_gemm: unexpected problem shape\n");
        return;
    }
    hipError_t e = hipFuncSetAttribute((const void*)ag_gemm, hipFuncAttributeMaxDynamicSharedMemorySize, LDS_BYTES);
    if (e != hipSuccess) { fprintf(stderr, "ag_gemm: hipFuncSetAttribute failed: %s\n", hipGetErrorString(e)); return; }
    Params p;
    p.x = (const unsigned char*)d_in[0]; p.w = (const unsigned char*)d_in[1];
    p.sx = (const float*)d_in[2]; p.sw = (const float*)d_in[3]; p.out = (float*)d_out;
    for (int i = 0; i < NR; ++i) p.ws[i] = (char*)d_ws_all[i];
    p.rank = rank; p.pad = 0;
    hipLaunchKernelGGL(ag_gemm, dim3(256), dim3(512), LDS_BYTES, stream, p);
    e = hipGetLastError();
    if (e != hipSuccess) fprintf(stderr, "ag_gemm: launch failed: %s\n", hipGetErrorString(e));
}
```

```cpp
#include <hip/hip_runtime.h>
#include <rccl/rccl.h>
#include <cstdio>

#define ST_MODE 2
#define EARLY_FENCE 2
#define EPI 1
#define LAG 0
#define ILV 1

namespace agg {
#define LAS __attribute__((address_space(3)))
#define GAS __attribute__((address_space(1)))
typedef int v8i __attribute__((ext_vector_type(8)));
typedef int v4i __attribute__((ext_vector_type(4)));
typedef int v2i __attribute__((ext_vector_type(2)));
typedef unsigned v4u __attribute__((ext_vector_type(4)));
typedef float v4f __attribute__((ext_vector_type(4)));

constexpr int NR = 8;
constexpr int MSH = 512, KD = 4096, NL = 2048;
constexpr int BM = 256, BN = 128, BK = 128, NT = KD / BK;
constexpr int A_ST = BM * BK, B_ST = BK * BN, ST = A_ST + B_ST, NSTAGE = 3;
constexpr int LDS_CTL = NSTAGE * ST;
constexpr int LDS_BYTES = LDS_CTL + 64;
constexpr size_t OFF_CNT = 0;
constexpr size_t OFF_TMO = 32768;
constexpr size_t OFF_FLAG = 65536;
constexpr size_t OFF_STG = 1u << 20;
constexpr size_t STG_BYTES = (size_t)MSH * KD;
constexpr size_t OFF_DIAG = 8u << 20;
constexpr size_t WS_NEED = 32u << 20;
constexpr unsigned long long DEADLINE_TICKS = 50000000ull;

struct Params {
    const unsigned char* x; const unsigned char* w; const float* sx; const float* sw; float* out;
    char* ws[NR];
    int rank; int pad;
};

__device__ __forceinline__ unsigned long long rt() { return __builtin_amdgcn_s_memrealtime(); }
template <int N> __device__ __forceinline__ void waitv() { asm volatile("s_waitcnt vmcnt(%0)" :: "n"(N) : "memory"); }
__device__ __forceinline__ void dma16(unsigned ldsaddr, unsigned voff, const void* base) {
    asm volatile("s_mov_b32 m0, %0\n\ts_nop 0\n\tglobal_load_lds_dwordx4 %1, %2" :: "s"(ldsaddr), "v"(voff), "s"(base) : "memory");
}
__device__ __forceinline__ const unsigned char* uni(const void* p) {
    const unsigned long long v = (unsigned long long)p;
    const unsigned lo = __builtin_amdgcn_readfirstlane((unsigned)v), hi = __builtin_amdgcn_readfirstlane((unsigned)(v >> 32));
    return (const unsigned char*)(((unsigned long long)hi << 32) | lo);
}
__device__ __forceinline__ void bar() { asm volatile("" ::: "memory"); __builtin_amdgcn_s_barrier(); asm volatile("" ::: "memory"); }

#define STAMP(i) do {} while (0)

__global__ __launch_bounds__(512, 2) void ag_gemm(Params p) {
    extern __shared__ __attribute__((aligned(16))) char smem[];
    LAS char* l = (LAS char*)smem;
    const unsigned lbase = (unsigned)(size_t)l;
    const int tid = threadIdx.x, lane = tid & 63;
    const int wid = __builtin_amdgcn_readfirstlane(tid >> 6);
    const int b = blockIdx.x, g = b & 7, ti = b >> 3, rtile = ti & 1, ct = ti >> 1;
    const int src = (p.rank + g) & 7;
    STAMP(0);
    char* const self = p.ws[0];
    char* const w1 = p.ws[1]; char* const w2 = p.ws[2]; char* const w3 = p.ws[3]; char* const w4 = p.ws[4];
    char* const w5 = p.ws[5]; char* const w6 = p.ws[6]; char* const w7 = p.ws[7];
    asm volatile("" :: "s"(self), "s"(w1), "s"(w2), "s"(w3), "s"(w4), "s"(w5), "s"(w6), "s"(w7), "s"(p.x), "s"(p.w));

    unsigned voffB[2];
#pragma unroll
    for (int j = 0; j < 2; ++j) {
        const int pb = wid * 2 + j, k = pb * 8 + (lane >> 3);
        const int ch = (lane & 7) ^ (((lane >> 4) & 3) | (((pb >> 1) & 1) << 2));
        voffB[j] = (unsigned)(k * NL + ct * BN + ch * 16);
    }
    auto issueB = [&](int kt, int slot) {
        const unsigned char* base = p.w + (size_t)kt * BK * NL;
#pragma unroll
        for (int j = 0; j < 2; ++j) dma16(lbase + slot * ST + A_ST + (wid * 2 + j) * 1024, voffB[j], base);
    };
    if (wid < 4) {
        const int skt = b >> 3, sch = tid & 7, srow = (b & 7) * 64 + (tid >> 3);
        const unsigned char* sp = p.x + (size_t)srow * KD + skt * BK + sch * 16;
        const v4u sv0 = *(const GAS v4u*)sp, sv1 = *(const GAS v4u*)(sp + 32 * KD);
        const unsigned cnt = __hip_atomic_load((GAS unsigned*)(self + OFF_CNT + b * 128), __ATOMIC_RELAXED, __HIP_MEMORY_SCOPE_AGENT);
        const unsigned c = __builtin_amdgcn_readfirstlane(cnt) + 1;
        char* dst = self + OFF_STG + (c & 1) * STG_BYTES + (size_t)skt * (MSH * BK) + srow * BK + sch * 16;
        asm volatile("global_store_dwordx4 %0, %1, off sc0 sc1\n\tglobal_store_dwordx4 %2, %3, off sc0 sc1\n\ts_nop 1" :: "v"(dst), "v"(sv0), "v"(dst + 32 * BK), "v"(sv1) : "memory");
        if (tid == 0) *(LAS unsigned*)(l + LDS_CTL + 4) = c;
        issueB(0, 0); issueB(1, 1);
        asm volatile("s_waitcnt vmcnt(4) lgkmcnt(0)" ::: "memory");
    } else {
        issueB(0, 0); issueB(1, 1);
    }
    bar();
    const unsigned call = __builtin_amdgcn_readfirstlane(*(LAS unsigned*)(l + LDS_CTL + 4));
    if (wid == 1) {
        if (lane == 0) {
            const unsigned fo = (unsigned)(OFF_FLAG + p.rank * 1024 + b * 4);
            char* const pw[7] = {w1, w2, w3, w4, w5, w6, w7};
#pragma unroll
            for (int q = 0; q < 7; ++q)
                __hip_atomic_store((GAS unsigned*)(pw[q] + fo), call, __ATOMIC_RELAXED, __HIP_MEMORY_SCOPE_SYSTEM);
            __hip_atomic_store((GAS unsigned*)(self + OFF_CNT + b * 128), call, __ATOMIC_RELAXED, __HIP_MEMORY_SCOPE_AGENT);
        }
    }
    STAMP(1);

    const unsigned char* Abase; unsigned RS, KS;
    const bool remote = g != 0;
    if (!remote) { Abase = p.x; RS = KD; KS = BK; }
    else { Abase = (const unsigned char*)(p.ws[g] + OFF_STG + (call & 1) * STG_BYTES); RS = BK; KS = MSH * BK; }
    Abase = uni(Abase);
    unsigned voffA[4];
#pragma unroll
    for (int j = 0; j < 4; ++j) {
        const int pa = wid * 4 + j, row = pa * 8 + (lane >> 3);
        const int ch = (lane & 7) ^ ((pa & 1) * 4 + (lane >> 4));
        voffA[j] = (unsigned)(rtile * BM + row) * RS + ch * 16;
    }
    auto issueA = [&](int kt, int slot) {
        const unsigned char* base = Abase + (size_t)kt * KS;
#pragma unroll
        for (int j = 0; j < 4; ++j) dma16(lbase + slot * ST + (wid * 4 + j) * 1024, voffA[j], base);
    };
    GAS unsigned* tmo = (GAS unsigned*)(self + OFF_TMO);
    bool ok = true;
    if (remote) {
        if (wid == 0) {
            const char* fp = self + OFF_FLAG + src * 1024 + lane * 16;
            const unsigned long long t0 = rt();
            unsigned good = 1;
            for (unsigned it = 1;; ++it) {
                v4u f;
                asm volatile("global_load_dwordx4 %0, %1, off sc0 sc1\n\ts_waitcnt vmcnt(0)" : "=&v"(f) : "v"(fp) : "memory");
                const bool all = f.x >= call && f.y >= call && f.z >= call && f.w >= call;
                if (__all(all)) break;
                if ((it & 15) == 0) {
                    if (__builtin_amdgcn_readfirstlane(__hip_atomic_load(tmo, __ATOMIC_RELAXED, __HIP_MEMORY_SCOPE_AGENT)) == call || rt() - t0 > DEADLINE_TICKS) {
                        if (lane == 0) __hip_atomic_store(tmo, call, __ATOMIC_RELAXED, __HIP_MEMORY_SCOPE_AGENT);
                        good = 0; break;
                    }
                    __builtin_amdgcn_s_sleep(2);
                }
            }
            STAMP(10);
            STAMP(11);
            if (lane == 0) *(LAS unsigned*)(l + LDS_CTL) = good;
            asm volatile("s_waitcnt lgkmcnt(0)" ::: "memory");
        }
        bar();
        ok = __builtin_amdgcn_readfirstlane(*(LAS unsigned*)(l + LDS_CTL)) != 0;
    }
    STAMP(2);
    if (ok) { issueA(0, 0); issueA(1, 1); }

    const int wm = wid >> 1, wn = wid & 1;
    const unsigned sA = (lane & 15) >> 1, g4 = lane >> 4;
    const unsigned aA0 = (unsigned)(wm * 64 + (lane & 15)) * 128 + ((g4 ^ sA) * 16), aA1 = aA0 ^ 64;
    const unsigned rB = (lane & 15) >> 1, sB = ((rB >> 1) & 3) | ((g4 & 1) << 2);
    unsigned aB[4];
#pragma unroll
    for (int ni = 0; ni < 4; ++ni) aB[ni] = A_ST + (16 * g4 + rB) * 128 + (((wn * 4 + ni) ^ sB) * 16) + (lane & 1) * 8;

    v4f acc[4][4];
#pragma unroll
    for (int mi = 0; mi < 4; ++mi)
#pragma unroll
        for (int ni = 0; ni < 4; ++ni) acc[mi][ni] = (v4f){0.f, 0.f, 0.f, 0.f};

    unsigned tmov = 0;
    if (ok) {
        waitv<4>();
        bar();
        STAMP(3);
        int slot = 0;
        for (int k = 0; k < NT; ++k) {
            const unsigned so = slot * ST;
            v8i Af[4], Bf[4];
#pragma unroll
            for (int ni = 0; ni < 4; ++ni) {
                LAS char* q = l + so + aB[ni];
                const v2i b0 = __builtin_amdgcn_ds_read_tr8_b64_v2i32((LAS v2i*)(q));
                const v2i b1 = __builtin_amdgcn_ds_read_tr8_b64_v2i32((LAS v2i*)(q + 1024));
                const v2i b2 = __builtin_amdgcn_ds_read_tr8_b64_v2i32((LAS v2i*)(q + 8192));
                const v2i b3 = __builtin_amdgcn_ds_read_tr8_b64_v2i32((LAS v2i*)(q + 9216));
                Bf[ni] = (v8i){b0.x, b0.y, b1.x, b1.y, b2.x, b2.y, b3.x, b3.y};
            }
#pragma unroll
            for (int mi = 0; mi < 4; ++mi) {
                const v4i a0 = *(LAS v4i*)(l + so + aA0 + mi * 2048);
                const v4i a1 = *(LAS v4i*)(l + so + aA1 + mi * 2048);
                Af[mi] = (v8i){a0.x, a0.y, a0.z, a0.w, a1.x, a1.y, a1.z, a1.w};
            }
            const int s2 = slot == 0 ? 2 : slot - 1;
            if (k == NT - 1) tmov = __hip_atomic_load(tmo, __ATOMIC_RELAXED, __HIP_MEMORY_SCOPE_AGENT);
            {
                const bool more = k + 2 < NT;
                const unsigned char* baseA = Abase + (size_t)(k + 2) * KS;
                const unsigned char* baseB = p.w + (size_t)(k + 2) * BK * NL;
                const unsigned la = lbase + s2 * ST + wid * 4096, lb = lbase + s2 * ST + A_ST + wid * 2048;
#pragma unroll
                for (int mi = 0; mi < 4; ++mi) {
#pragma unroll
                    for (int ni = 0; ni < 4; ++ni) {
                        acc[mi][ni] = __builtin_amdgcn_mfma_scale_f32_16x16x128_f8f6f4(Bf[ni], Af[mi], acc[mi][ni], 0, 0, 0, 0, 0, 0);
                        const int i = mi * 4 + ni;
                        if ((i & 1) == 1 && i < 12) {
                            const int j = i >> 1;
                            __builtin_amdgcn_sched_barrier(0);
                            if (more) { if (j < 4) dma16(la + j * 1024, voffA[j], baseA); else dma16(lb + (j - 4) * 1024, voffB[j - 4], baseB); }
                            __builtin_amdgcn_sched_barrier(0);
                        }
                    }
                }
            }
            asm volatile("s_waitcnt lgkmcnt(0)" ::: "memory");
            if (k + 2 < NT) waitv<6>(); else waitv<0>();
            if (k + 1 < NT) bar();
            slot = slot == 2 ? 0 : slot + 1;
        }
        ok = __builtin_amdgcn_readfirstlane(tmov) != call;
    } else {
        waitv<0>();
    }
    STAMP(8);
    const float sc = ok ? p.sx[0] * p.sw[0] : __builtin_nanf("");
    const int orow = src * MSH + rtile * BM + wm * 64 + (lane & 7), ocol = ct * BN + wn * 64 + (lane & 8) * 2 + 4 * (lane >> 4);
    float* optr = p.out + (size_t)orow * NL + ocol;
#pragma unroll
    for (int mi = 0; mi < 4; ++mi)
#pragma unroll
        for (int np = 0; np < 2; ++np) {
            const v4f a = acc[mi][2 * np] * sc, bb = acc[mi][2 * np + 1] * sc;
            v4f x, y;
#pragma unroll
            for (int j = 0; j < 4; ++j) {
                const int aj = __float_as_int(a[j]), bj = __float_as_int(bb[j]);
                x[j] = __int_as_float(__builtin_amdgcn_update_dpp(aj, bj, 0x128, 0xf, 0xc, false));
                y[j] = __int_as_float(__builtin_amdgcn_update_dpp(bj, aj, 0x128, 0xf, 0x3, false));
            }
            float* q = optr + (size_t)mi * 16 * NL + np * 32;
            asm volatile("global_store_dwordx4 %0, %1, off sc1\n\ts_nop 1" :: "v"(q), "v"(x) : "memory");
            asm volatile("global_store_dwordx4 %0, %1, off sc1\n\ts_nop 1" :: "v"(q + 8 * NL), "v"(y) : "memory");
        }
}
}

extern "C" void kernel_launch(int rank, int nranks, ncclComm_t comm, hipStream_t stream,
                              void* const* d_in, const int* in_sizes, int n_in,
                              void* d_out, int out_size, const int* in_dtypes, int out_dtype,
                              void* const* d_out_all, void* const* d_ws_all, size_t ws_bytes) {
    using namespace agg;
    (void)comm; (void)in_dtypes; (void)out_dtype; (void)d_out_all;
    if (nranks != NR || n_in != 4 || (unsigned)rank >= (unsigned)NR || ws_bytes < WS_NEED ||
        in_sizes[0] != MSH * KD || in_sizes[1] != KD * NL || out_size != NR * MSH * NL) {
        fprintf(stderr, "ag_gemm: unexpected problem shape\n");
        return;
    }
    hipError_t e = hipFuncSetAttribute((const void*)ag_gemm, hipFuncAttributeMaxDynamicSharedMemorySize, LDS_BYTES);
    if (e != hipSuccess) { fprintf(stderr, "ag_gemm: hipFuncSetAttribute failed: %s\n", hipGetErrorString(e)); return; }
    Params p;
    p.x = (const unsigned char*)d_in[0]; p.w = (const unsigned char*)d_in[1];
    p.sx = (const float*)d_in[2]; p.sw = (const float*)d_in[3]; p.out = (float*)d_out;
    for (int i = 0; i < NR; ++i) p.ws[i] = (char*)d_ws_all[(rank + i) & 7];
    p.rank = rank; p.pad = 0;
    hipLaunchKernelGGL(ag_gemm, dim3(256), dim3(512), LDS_BYTES, stream, p);
    e = hipGetLastError();
    if (e != hipSuccess) fprintf(stderr, "ag_gemm: launch failed: %s\n", hipGetErrorString(e));
}
```

```cpp
#include <hip/hip_runtime.h>
#include <rccl/rccl.h>
#include <cstdio>

#define ST_MODE 2
#define EARLY_FENCE 2
#define EPI 1
#define POLL_GAP 5
#define LAG 0
#define ILV 1

namespace agg {
#define LAS __attribute__((address_space(3)))
#define GAS __attribute__((address_space(1)))
typedef int v8i __attribute__((ext_vector_type(8)));
typedef int v4i __attribute__((ext_vector_type(4)));
typedef int v2i __attribute__((ext_vector_type(2)));
typedef unsigned v4u __attribute__((ext_vector_type(4)));
typedef float v4f __attribute__((ext_vector_type(4)));

constexpr int NR = 8;
constexpr int MSH = 512, KD = 4096, NL = 2048;
constexpr int BM = 256, BN = 128, BK = 128, NT = KD / BK;
constexpr int A_ST = BM * BK, B_ST = BK * BN, ST = A_ST + B_ST, NSTAGE = 3;
constexpr int LDS_POLL = NSTAGE * ST;
constexpr int LDS_FLAGS = LDS_POLL + 1024;
constexpr int LDS_CTL = LDS_FLAGS + 1024;
constexpr int LDS_BYTES = LDS_CTL + 64;
constexpr size_t OFF_CNT = 0;
constexpr size_t OFF_TMO = 32768;
constexpr size_t OFF_FLAG = 65536;
constexpr size_t OFF_STG = 1u << 20;
constexpr size_t STG_BYTES = (size_t)MSH * KD;
constexpr size_t OFF_DIAG = 8u << 20;
constexpr size_t WS_NEED = 32u << 20;
constexpr unsigned long long DEADLINE_TICKS = 50000000ull;

struct Params {
    const unsigned char* x; const unsigned char* w; const float* sx; const float* sw; float* out;
    char* ws[NR];
    int rank; int pad;
};

__device__ __forceinline__ unsigned long long rt() { return __builtin_amdgcn_s_memrealtime(); }
template <int N> __device__ __forceinline__ void waitv() { asm volatile("s_waitcnt vmcnt(%0)" :: "n"(N) : "memory"); }
__device__ __forceinline__ void dma16(unsigned ldsaddr, unsigned voff, const void* base) {
    asm volatile("s_mov_b32 m0, %0\n\ts_nop 0\n\tglobal_load_lds_dwordx4 %1, %2" :: "s"(ldsaddr), "v"(voff), "s"(base) : "memory");
}
__device__ __forceinline__ void dma4_sys(unsigned ldsaddr, unsigned voff, const void* base) {
    asm volatile("s_mov_b32 m0, %0\n\ts_nop 0\n\tglobal_load_lds_dword %1, %2 sc0 sc1" :: "s"(ldsaddr), "v"(voff), "s"(base) : "memory");
}
__device__ __forceinline__ void dma16_sys(unsigned ldsaddr, unsigned voff, const void* base) {
    asm volatile("s_mov_b32 m0, %0\n\ts_nop 0\n\tglobal_load_lds_dwordx4 %1, %2 sc0 sc1" :: "s"(ldsaddr), "v"(voff), "s"(base) : "memory");
}
__device__ __forceinline__ const unsigned char* uni(const void* p) {
    const unsigned long long v = (unsigned long long)p;
    const unsigned lo = __builtin_amdgcn_readfirstlane((unsigned)v), hi = __builtin_amdgcn_readfirstlane((unsigned)(v >> 32));
    return (const unsigned char*)(((unsigned long long)hi << 32) | lo);
}
__device__ __forceinline__ void bar() { asm volatile("" ::: "memory"); __builtin_amdgcn_s_barrier(); asm volatile("" ::: "memory"); }

#define STAMP(i) do {} while (0)

__global__ __launch_bounds__(512, 2) void ag_gemm(Params p) {
    extern __shared__ __attribute__((aligned(16))) char smem[];
    LAS char* l = (LAS char*)smem;
    const unsigned lbase = (unsigned)(size_t)l;
    const int tid = threadIdx.x, lane = tid & 63;
    const int wid = __builtin_amdgcn_readfirstlane(tid >> 6);
    const int b = blockIdx.x, g = b & 7, ti = b >> 3, rtile = ti & 1, ct = ti >> 1;
    const int src = (p.rank + g) & 7;
    STAMP(0);
    char* const self = p.ws[0];
    char* const w1 = p.ws[1]; char* const w2 = p.ws[2]; char* const w3 = p.ws[3]; char* const w4 = p.ws[4];
    char* const w5 = p.ws[5]; char* const w6 = p.ws[6]; char* const w7 = p.ws[7];
    asm volatile("" :: "s"(self), "s"(w1), "s"(w2), "s"(w3), "s"(w4), "s"(w5), "s"(w6), "s"(w7), "s"(p.x), "s"(p.w));
    typedef const __attribute__((address_space(4))) float* cfp;
    float scale = *(cfp)p.sx * *(cfp)p.sw;

    unsigned voffB[2];
#pragma unroll
    for (int j = 0; j < 2; ++j) {
        const int pb = wid * 2 + j, k = pb * 8 + (lane >> 3);
        const int ch = (lane & 7) ^ (((lane >> 4) & 3) | (((pb >> 1) & 1) << 2));
        voffB[j] = (unsigned)(k * NL + ct * BN + ch * 16);
    }
    auto issueB = [&](int kt, int slot) {
        const unsigned char* base = p.w + (size_t)kt * BK * NL;
#pragma unroll
        for (int j = 0; j < 2; ++j) dma16(lbase + slot * ST + A_ST + (wid * 2 + j) * 1024, voffB[j], base);
    };
    if (wid < 4) {
        const int skt = b >> 3, sch = tid & 7, srow = (b & 7) * 64 + (tid >> 3);
        const unsigned char* sp = p.x + (size_t)srow * KD + skt * BK + sch * 16;
        const v4u sv0 = *(const GAS v4u*)sp, sv1 = *(const GAS v4u*)(sp + 32 * KD);
        const unsigned cnt = __hip_atomic_load((GAS unsigned*)(self + OFF_CNT + b * 128), __ATOMIC_RELAXED, __HIP_MEMORY_SCOPE_AGENT);
        const unsigned c = __builtin_amdgcn_readfirstlane(cnt) + 1;
        char* dst = self + OFF_STG + (c & 1) * STG_BYTES + (size_t)skt * (MSH * BK) + srow * BK + sch * 16;
        asm volatile("global_store_dwordx4 %0, %1, off sc0 sc1\n\tglobal_store_dwordx4 %2, %3, off sc0 sc1\n\ts_nop 1" :: "v"(dst), "v"(sv0), "v"(dst + 32 * BK), "v"(sv1) : "memory");
        if (tid == 0) *(LAS unsigned*)(l + LDS_CTL + 4) = c;
        issueB(0, 0); issueB(1, 1);
        asm volatile("s_waitcnt vmcnt(4) lgkmcnt(0)" ::: "memory");
    } else {
        issueB(0, 0); issueB(1, 1);
    }
    bar();
    const unsigned call = __builtin_amdgcn_readfirstlane(*(LAS unsigned*)(l + LDS_CTL + 4));
    if (wid == 1) {
        if (lane == 0) {
            const unsigned fo = (unsigned)(OFF_FLAG + p.rank * 1024 + b * 4);
            char* const pw[7] = {w1, w2, w3, w4, w5, w6, w7};
#pragma unroll
            for (int q = 0; q < 7; ++q)
                __hip_atomic_store((GAS unsigned*)(pw[q] + fo), call, __ATOMIC_RELAXED, __HIP_MEMORY_SCOPE_SYSTEM);
            __hip_atomic_store((GAS unsigned*)(self + OFF_CNT + b * 128), call, __ATOMIC_RELAXED, __HIP_MEMORY_SCOPE_AGENT);
        }
    }
    STAMP(1);

    const unsigned char* Abase; unsigned RS, KS;
    const bool remote = g != 0;
    if (!remote) { Abase = p.x; RS = KD; KS = BK; }
    else { Abase = (const unsigned char*)(p.ws[g] + OFF_STG + (call & 1) * STG_BYTES); RS = BK; KS = MSH * BK; }
    Abase = uni(Abase);
    unsigned voffA[4];
#pragma unroll
    for (int j = 0; j < 4; ++j) {
        const int pa = wid * 4 + j, row = pa * 8 + (lane >> 3);
        const int ch = (lane & 7) ^ ((pa & 1) * 4 + (lane >> 4));
        voffA[j] = (unsigned)(rtile * BM + row) * RS + ch * 16;
    }
    auto issueA = [&](int kt, int slot) {
        const unsigned char* base = Abase + (size_t)kt * KS;
#pragma unroll
        for (int j = 0; j < 4; ++j) dma16(lbase + slot * ST + (wid * 4 + j) * 1024, voffA[j], base);
    };
    GAS unsigned* tmo = (GAS unsigned*)(self + OFF_TMO);
    bool ok = true;
    const unsigned char* fbase = uni(self + OFF_FLAG + src * 1024);
    auto wait_all = [&]() -> unsigned {
        const char* fp = (const char*)fbase + lane * 16;
        const unsigned long long t0 = rt();
        for (unsigned it = 1;; ++it) {
            v4u f;
            asm volatile("global_load_dwordx4 %0, %1, off sc0 sc1\n\ts_waitcnt vmcnt(0)" : "=&v"(f) : "v"(fp) : "memory");
            if (__all(f.x >= call && f.y >= call && f.z >= call && f.w >= call)) return 1;
            if ((it & 15) == 0) {
                if (__builtin_amdgcn_readfirstlane(__hip_atomic_load(tmo, __ATOMIC_RELAXED, __HIP_MEMORY_SCOPE_AGENT)) == call || rt() - t0 > DEADLINE_TICKS) {
                    if (lane == 0) __hip_atomic_store(tmo, call, __ATOMIC_RELAXED, __HIP_MEMORY_SCOPE_AGENT);
                    return 0;
                }
                __builtin_amdgcn_s_sleep(2);
            }
        }
    };
    if (remote) {
        if (wid == 0) {
            const unsigned pv = (lane & 15) * 4;
            const unsigned long long t0 = rt();
            unsigned good = 1;
#pragma unroll
            for (int i = 0; i < 3; ++i) { dma4_sys(lbase + LDS_POLL + i * 256, pv, fbase); __builtin_amdgcn_s_sleep(POLL_GAP); }
            for (unsigned it = 0;; ++it) {
                dma4_sys(lbase + LDS_POLL + ((it + 3) & 3) * 256, pv, fbase);
                waitv<3>();
                const unsigned f = *(volatile LAS unsigned*)(l + LDS_POLL + (it & 3) * 256 + lane * 4);
                if (__all(f >= call)) break;
                if ((it & 63) == 63) {
                    if (__builtin_amdgcn_readfirstlane(__hip_atomic_load(tmo, __ATOMIC_RELAXED, __HIP_MEMORY_SCOPE_AGENT)) == call || rt() - t0 > DEADLINE_TICKS) {
                        if (lane == 0) __hip_atomic_store(tmo, call, __ATOMIC_RELAXED, __HIP_MEMORY_SCOPE_AGENT);
                        good = 0; break;
                    }
                }
            }
            STAMP(10);
            if (lane == 0) *(LAS unsigned*)(l + LDS_CTL) = good;
            asm volatile("s_waitcnt lgkmcnt(0)" ::: "memory");
        }
        bar();
        ok = __builtin_amdgcn_readfirstlane(*(LAS unsigned*)(l + LDS_CTL)) != 0;
    }
    STAMP(2);
    if (ok) {
        issueA(0, 0);
        if (remote && wid == 0) dma16_sys(lbase + LDS_FLAGS, lane * 16, fbase);
        issueA(1, 1);
    }

    const int wm = wid >> 1, wn = wid & 1;
    const unsigned sA = (lane & 15) >> 1, g4 = lane >> 4;
    const unsigned aA0 = (unsigned)(wm * 64 + (lane & 15)) * 128 + ((g4 ^ sA) * 16), aA1 = aA0 ^ 64;
    const unsigned rB = (lane & 15) >> 1, sB = ((rB >> 1) & 3) | ((g4 & 1) << 2);
    unsigned aB[4];
#pragma unroll
    for (int ni = 0; ni < 4; ++ni) aB[ni] = A_ST + (16 * g4 + rB) * 128 + (((wn * 4 + ni) ^ sB) * 16) + (lane & 1) * 8;

    v4f acc[4][4];
#pragma unroll
    for (int mi = 0; mi < 4; ++mi)
#pragma unroll
        for (int ni = 0; ni < 4; ++ni) acc[mi][ni] = (v4f){0.f, 0.f, 0.f, 0.f};

    unsigned tmov = 0;
    if (ok) {
        waitv<4>();
        bar();
        if (remote) {
            const v4u f = *(LAS v4u*)(l + LDS_FLAGS + lane * 16);
            if (!__all(f.x >= call && f.y >= call && f.z >= call && f.w >= call)) {
                if (wid == 0) { const unsigned g2 = wait_all(); if (lane == 0) *(LAS unsigned*)(l + LDS_CTL) = g2; asm volatile("s_waitcnt lgkmcnt(0)" ::: "memory"); }
                bar();
                ok = __builtin_amdgcn_readfirstlane(*(LAS unsigned*)(l + LDS_CTL)) != 0;
            }
        }
    }
    if (ok) {
        STAMP(3);
        int slot = 0;
        for (int k = 0; k < NT; ++k) {
            const unsigned so = slot * ST;
            v8i Af[4], Bf[4];
#pragma unroll
            for (int ni = 0; ni < 4; ++ni) {
                LAS char* q = l + so + aB[ni];
                const v2i b0 = __builtin_amdgcn_ds_read_tr8_b64_v2i32((LAS v2i*)(q));
                const v2i b1 = __builtin_amdgcn_ds_read_tr8_b64_v2i32((LAS v2i*)(q + 1024));
                const v2i b2 = __builtin_amdgcn_ds_read_tr8_b64_v2i32((LAS v2i*)(q + 8192));
                const v2i b3 = __builtin_amdgcn_ds_read_tr8_b64_v2i32((LAS v2i*)(q + 9216));
                Bf[ni] = (v8i){b0.x, b0.y, b1.x, b1.y, b2.x, b2.y, b3.x, b3.y};
            }
#pragma unroll
            for (int mi = 0; mi < 4; ++mi) {
                const v4i a0 = *(LAS v4i*)(l + so + aA0 + mi * 2048);
                const v4i a1 = *(LAS v4i*)(l + so + aA1 + mi * 2048);
                Af[mi] = (v8i){a0.x, a0.y, a0.z, a0.w, a1.x, a1.y, a1.z, a1.w};
            }
            const int s2 = slot == 0 ? 2 : slot - 1;
            if (k == NT - 1) tmov = __hip_atomic_load(tmo, __ATOMIC_RELAXED, __HIP_MEMORY_SCOPE_AGENT);
            {
                const bool more = k + 2 < NT;
                const unsigned char* baseA = Abase + (size_t)(k + 2) * KS;
                const unsigned char* baseB = p.w + (size_t)(k + 2) * BK * NL;
                const unsigned la = lbase + s2 * ST + wid * 4096, lb = lbase + s2 * ST + A_ST + wid * 2048;
#pragma unroll
                for (int mi = 0; mi < 4; ++mi) {
#pragma unroll
                    for (int ni = 0; ni < 4; ++ni) {
                        acc[mi][ni] = __builtin_amdgcn_mfma_scale_f32_16x16x128_f8f6f4(Bf[ni], Af[mi], acc[mi][ni], 0, 0, 0, 0, 0, 0);
                        const int i = mi * 4 + ni;
                        if ((i & 1) == 1 && i < 12) {
                            const int j = i >> 1;
                            __builtin_amdgcn_sched_barrier(0);
                            if (more) { if (j < 4) dma16(la + j * 1024, voffA[j], baseA); else dma16(lb + (j - 4) * 1024, voffB[j - 4], baseB); }
                            __builtin_amdgcn_sched_barrier(0);
                        }
                    }
                }
            }
            asm volatile("s_waitcnt lgkmcnt(0)" ::: "memory");
            if (k + 2 < NT) waitv<6>(); else waitv<0>();
            if (k + 1 < NT) bar();
            slot = slot == 2 ? 0 : slot + 1;
        }
        ok = __builtin_amdgcn_readfirstlane(tmov) != call;
    } else {
        waitv<0>();
    }
    STAMP(8);
    const float sc = ok ? scale : __builtin_nanf("");
    const int orow = src * MSH + rtile * BM + wm * 64 + (lane & 7), ocol = ct * BN + wn * 64 + (lane & 8) * 2 + 4 * (lane >> 4);
    float* optr = p.out + (size_t)orow * NL + ocol;
#pragma unroll
    for (int mi = 0; mi < 4; ++mi)
#pragma unroll
        for (int np = 0; np < 2; ++np) {
            const v4f a = acc[mi][2 * np] * sc, bb = acc[mi][2 * np + 1] * sc;
            v4f x, y;
#pragma unroll
            for (int j = 0; j < 4; ++j) {
                const int aj = __float_as_int(a[j]), bj = __float_as_int(bb[j]);
                x[j] = __int_as_float(__builtin_amdgcn_update_dpp(aj, bj, 0x128, 0xf, 0xc, false));
                y[j] = __int_as_float(__builtin_amdgcn_update_dpp(bj, aj, 0x128, 0xf, 0x3, false));
            }
            float* q = optr + (size_t)mi * 16 * NL + np * 32;
            asm volatile("global_store_dwordx4 %0, %1, off sc1\n\ts_nop 1" :: "v"(q), "v"(x) : "memory");
            asm volatile("global_store_dwordx4 %0, %1, off sc1\n\ts_nop 1" :: "v"(q + 8 * NL), "v"(y) : "memory");
        }
}
}

extern "C" void kernel_launch(int rank, int nranks, ncclComm_t comm, hipStream_t stream,
                              void* const* d_in, const int* in_sizes, int n_in,
                              void* d_out, int out_size, const int* in_dtypes, int out_dtype,
                              void* const* d_out_all, void* const* d_ws_all, size_t ws_bytes) {
    using namespace agg;
    (void)comm; (void)in_dtypes; (void)out_dtype; (void)d_out_all;
    if (nranks != NR || n_in != 4 || (unsigned)rank >= (unsigned)NR || ws_bytes < WS_NEED ||
        in_sizes[0] != MSH * KD || in_sizes[1] != KD * NL || out_size != NR * MSH * NL) {
        fprintf(stderr, "ag_gemm: unexpected problem shape\n");
        return;
    }
    hipError_t e = hipFuncSetAttribute((const void*)ag_gemm, hipFuncAttributeMaxDynamicSharedMemorySize, LDS_BYTES);
    if (e != hipSuccess) { fprintf(stderr, "ag_gemm: hipFuncSetAttribute failed: %s\n", hipGetErrorString(e)); return; }
    Params p;
    p.x = (const unsigned char*)d_in[0]; p.w = (const unsigned char*)d_in[1];
    p.sx = (const float*)d_in[2]; p.sw = (const float*)d_in[3]; p.out = (float*)d_out;
    for (int i = 0; i < NR; ++i) p.ws[i] = (char*)d_ws_all[(rank + i) & 7];
    p.rank = rank; p.pad = 0;
    hipLaunchKernelGGL(ag_gemm, dim3(256), dim3(512), LDS_BYTES, stream, p);
    e = hipGetLastError();
    if (e != hipSuccess) fprintf(stderr, "ag_gemm: launch failed: %s\n", hipGetErrorString(e));
}
```

```cpp
#include <hip/hip_runtime.h>
#include <rccl/rccl.h>
#include <cstdio>
#include <cstdlib>

namespace agg {
#define LAS __attribute__((address_space(3)))
#define GAS __attribute__((address_space(1)))
typedef int v8i __attribute__((ext_vector_type(8)));
typedef int v4i __attribute__((ext_vector_type(4)));
typedef int v2i __attribute__((ext_vector_type(2)));
typedef unsigned v4u __attribute__((ext_vector_type(4)));
typedef float v4f __attribute__((ext_vector_type(4)));

constexpr int NR = 8;
constexpr int MSH = 512, KD = 4096, NL = 2048;
constexpr int BM = 256, BN = 128, BK = 128, NT = KD / BK;
constexpr int A_ST = BM * BK, B_ST = BK * BN, ST = A_ST + B_ST, NSTAGE = 3;
constexpr int LDS_POLL = NSTAGE * ST;
constexpr int LDS_FLAGS = LDS_POLL + 1024;
constexpr int LDS_CTL = LDS_FLAGS + 1024;
constexpr int LDS_BYTES = LDS_CTL + 64;
constexpr size_t OFF_CNT = 0;
constexpr size_t OFF_TMO = 32768;
constexpr size_t OFF_FLAG = 65536;
constexpr size_t OFF_STG = 1u << 20;
constexpr size_t STG_BYTES = (size_t)MSH * KD;
constexpr size_t OFF_DIAG = 8u << 20;
constexpr size_t WS_NEED = 32u << 20;
constexpr unsigned long long DEADLINE_TICKS = 50000000ull;

struct Params {
    const unsigned char* x; const unsigned char* w; const float* sx; const float* sw; float* out;
    char* ws[NR];
    int rank; unsigned perm;
};

__device__ __forceinline__ unsigned long long rt() { return __builtin_amdgcn_s_memrealtime(); }
template <int N> __device__ __forceinline__ void waitv() { asm volatile("s_waitcnt vmcnt(%0)" :: "n"(N) : "memory"); }
__device__ __forceinline__ void dma16(unsigned ldsaddr, unsigned voff, const void* base) {
    asm volatile("s_mov_b32 m0, %0\n\ts_nop 0\n\tglobal_load_lds_dwordx4 %1, %2" :: "s"(ldsaddr), "v"(voff), "s"(base) : "memory");
}
__device__ __forceinline__ void dma4_sys(unsigned ldsaddr, unsigned voff, const void* base) {
    asm volatile("s_mov_b32 m0, %0\n\ts_nop 0\n\tglobal_load_lds_dword %1, %2 sc0 sc1" :: "s"(ldsaddr), "v"(voff), "s"(base) : "memory");
}
__device__ __forceinline__ void dma16_sys(unsigned ldsaddr, unsigned voff, const void* base) {
    asm volatile("s_mov_b32 m0, %0\n\ts_nop 0\n\tglobal_load_lds_dwordx4 %1, %2 sc0 sc1" :: "s"(ldsaddr), "v"(voff), "s"(base) : "memory");
}
__device__ __forceinline__ const unsigned char* uni(const void* p) {
    const unsigned long long v = (unsigned long long)p;
    const unsigned lo = __builtin_amdgcn_readfirstlane((unsigned)v), hi = __builtin_amdgcn_readfirstlane((unsigned)(v >> 32));
    return (const unsigned char*)(((unsigned long long)hi << 32) | lo);
}
__device__ __forceinline__ void bar() { asm volatile("" ::: "memory"); __builtin_amdgcn_s_barrier(); asm volatile("" ::: "memory"); }

#define STAMP(i) do {} while (0)

__global__ __launch_bounds__(512, 2) void ag_gemm(Params p) {
    extern __shared__ __attribute__((aligned(16))) char smem[];
    LAS char* l = (LAS char*)smem;
    const unsigned lbase = (unsigned)(size_t)l;
    const int tid = threadIdx.x, lane = tid & 63;
    const int wid = __builtin_amdgcn_readfirstlane(tid >> 6);
    const int b = blockIdx.x, g = b & 7, ti = b >> 3, rtile = ti & 1, ct = ti >> 1;
    const int q = (p.perm >> (4 * g)) & 7;
    const int src = (p.rank + q) & 7;
    STAMP(0);
    char* const self = p.ws[0];
    char* const w1 = p.ws[1]; char* const w2 = p.ws[2]; char* const w3 = p.ws[3]; char* const w4 = p.ws[4];
    char* const w5 = p.ws[5]; char* const w6 = p.ws[6]; char* const w7 = p.ws[7];
    asm volatile("" :: "s"(self), "s"(w1), "s"(w2), "s"(w3), "s"(w4), "s"(w5), "s"(w6), "s"(w7), "s"(p.x), "s"(p.w));
    typedef const __attribute__((address_space(4))) float* cfp;
    const float scale = *(cfp)p.sx * *(cfp)p.sw;

    unsigned voffB[2];
#pragma unroll
    for (int j = 0; j < 2; ++j) {
        const int pb = wid * 2 + j, k = pb * 8 + (lane >> 3);
        const int ch = (lane & 7) ^ (((lane >> 4) & 3) | (((pb >> 1) & 1) << 2));
        voffB[j] = (unsigned)(k * NL + ct * BN + ch * 16);
    }
    auto issueB = [&](int kt, int slot) {
        const unsigned char* base = p.w + (size_t)kt * BK * NL;
#pragma unroll
        for (int j = 0; j < 2; ++j) dma16(lbase + slot * ST + A_ST + (wid * 2 + j) * 1024, voffB[j], base);
    };

    if (wid < 4) {
        const int skt = b >> 3, sch = tid & 7, srow = (b & 7) * 64 + (tid >> 3);
        const unsigned char* sp = p.x + (size_t)srow * KD + skt * BK + sch * 16;
        const v4u sv0 = *(const GAS v4u*)sp, sv1 = *(const GAS v4u*)(sp + 32 * KD);
        const unsigned cnt = __hip_atomic_load((GAS unsigned*)(self + OFF_CNT + b * 128), __ATOMIC_RELAXED, __HIP_MEMORY_SCOPE_AGENT);
        const unsigned c = __builtin_amdgcn_readfirstlane(cnt) + 1;
        char* dst = self + OFF_STG + (c & 1) * STG_BYTES + (size_t)skt * (MSH * BK) + srow * BK + sch * 16;
        asm volatile("global_store_dwordx4 %0, %1, off sc0 sc1\n\tglobal_store_dwordx4 %2, %3, off sc0 sc1\n\ts_nop 1" :: "v"(dst), "v"(sv0), "v"(dst + 32 * BK), "v"(sv1) : "memory");
        if (tid == 0) *(LAS unsigned*)(l + LDS_CTL + 4) = c;
        issueB(0, 0); issueB(1, 1);
        asm volatile("s_waitcnt vmcnt(4) lgkmcnt(0)" ::: "memory");
    } else {
        issueB(0, 0); issueB(1, 1);
    }
    bar();
    const unsigned call = __builtin_amdgcn_readfirstlane(*(LAS unsigned*)(l + LDS_CTL + 4));
    if (wid == 1 && lane == 0) {
        const unsigned fo = (unsigned)(OFF_FLAG + p.rank * 1024 + b * 4);
        char* const pw[7] = {w1, w2, w3, w4, w5, w6, w7};
#pragma unroll
        for (int q = 0; q < 7; ++q)
            __hip_atomic_store((GAS unsigned*)(pw[q] + fo), call, __ATOMIC_RELAXED, __HIP_MEMORY_SCOPE_SYSTEM);
        __hip_atomic_store((GAS unsigned*)(self + OFF_CNT + b * 128), call, __ATOMIC_RELAXED, __HIP_MEMORY_SCOPE_AGENT);
    }
    STAMP(1);

    const unsigned char* Abase; unsigned RS, KS;
    const bool remote = q != 0;
    if (!remote) { Abase = p.x; RS = KD; KS = BK; }
    else { Abase = (const unsigned char*)(p.ws[q] + OFF_STG + (call & 1) * STG_BYTES); RS = BK; KS = MSH * BK; }
    Abase = uni(Abase);
    unsigned voffA[4];
#pragma unroll
    for (int j = 0; j < 4; ++j) {
        const int pa = wid * 4 + j, row = pa * 8 + (lane >> 3);
        const int ch = (lane & 7) ^ ((pa & 1) * 4 + (lane >> 4));
        voffA[j] = (unsigned)(rtile * BM + row) * RS + ch * 16;
    }
    auto issueA = [&](int kt, int slot) {
        const unsigned char* base = Abase + (size_t)kt * KS;
#pragma unroll
        for (int j = 0; j < 4; ++j) dma16(lbase + slot * ST + (wid * 4 + j) * 1024, voffA[j], base);
    };
    GAS unsigned* tmo = (GAS unsigned*)(self + OFF_TMO);
    const unsigned char* fbase = uni(self + OFF_FLAG + src * 1024);
    auto gave_up = [&](unsigned long long t0) -> bool {
        if (__builtin_amdgcn_readfirstlane(__hip_atomic_load(tmo, __ATOMIC_RELAXED, __HIP_MEMORY_SCOPE_AGENT)) != call && rt() - t0 <= DEADLINE_TICKS) return false;
        if (lane == 0) __hip_atomic_store(tmo, call, __ATOMIC_RELAXED, __HIP_MEMORY_SCOPE_AGENT);
        return true;
    };
    bool ok = true;
    if (remote) {
        if (wid == 0) {
            const unsigned pv = (lane & 15) * 4;
            const unsigned long long t0 = rt();
            unsigned good = 1;
#pragma unroll
            for (int i = 0; i < 3; ++i) { dma4_sys(lbase + LDS_POLL + i * 256, pv, fbase); __builtin_amdgcn_s_sleep(5); }
            for (unsigned it = 0;; ++it) {
                dma4_sys(lbase + LDS_POLL + ((it + 3) & 3) * 256, pv, fbase);
                waitv<3>();
                const unsigned f = *(volatile LAS unsigned*)(l + LDS_POLL + (it & 3) * 256 + lane * 4);
                if (__all(f >= call)) break;
                if ((it & 63) == 63 && gave_up(t0)) { good = 0; break; }
            }
            if (lane == 0) *(LAS unsigned*)(l + LDS_CTL) = good;
            asm volatile("s_waitcnt lgkmcnt(0)" ::: "memory");
            STAMP(10);
        }
        bar();
        ok = __builtin_amdgcn_readfirstlane(*(LAS unsigned*)(l + LDS_CTL)) != 0;
    }
    STAMP(2);
    if (ok) {
        issueA(0, 0);
        if (remote && wid == 0) dma16_sys(lbase + LDS_FLAGS, lane * 16, fbase);
        issueA(1, 1);
    }

    const int wm = wid >> 1, wn = wid & 1;
    const unsigned sA = (lane & 15) >> 1, g4 = lane >> 4;
    const unsigned aA0 = (unsigned)(wm * 64 + (lane & 15)) * 128 + ((g4 ^ sA) * 16), aA1 = aA0 ^ 64;
    const unsigned rB = (lane & 15) >> 1, sB = ((rB >> 1) & 3) | ((g4 & 1) << 2);
    unsigned aB[4];
#pragma unroll
    for (int ni = 0; ni < 4; ++ni) aB[ni] = A_ST + (16 * g4 + rB) * 128 + (((wn * 4 + ni) ^ sB) * 16) + (lane & 1) * 8;

    v4f acc[4][4];
#pragma unroll
    for (int mi = 0; mi < 4; ++mi)
#pragma unroll
        for (int ni = 0; ni < 4; ++ni) acc[mi][ni] = (v4f){0.f, 0.f, 0.f, 0.f};

    if (ok) {
        waitv<4>();
        bar();
        if (remote) {
            const v4u f = *(LAS v4u*)(l + LDS_FLAGS + lane * 16);
            if (!__all(f.x >= call && f.y >= call && f.z >= call && f.w >= call)) {
                if (wid == 0) {
                    const char* fp = (const char*)fbase + lane * 16;
                    const unsigned long long t0 = rt();
                    unsigned good = 1;
                    for (unsigned it = 1;; ++it) {
                        v4u h;
                        asm volatile("global_load_dwordx4 %0, %1, off sc0 sc1\n\ts_waitcnt vmcnt(0)" : "=&v"(h) : "v"(fp) : "memory");
                        if (__all(h.x >= call && h.y >= call && h.z >= call && h.w >= call)) break;
                        if ((it & 15) == 0 && gave_up(t0)) { good = 0; break; }
                        __builtin_amdgcn_s_sleep(2);
                    }
                    if (lane == 0) *(LAS unsigned*)(l + LDS_CTL) = good;
                    asm volatile("s_waitcnt lgkmcnt(0)" ::: "memory");
                }
                bar();
                ok = __builtin_amdgcn_readfirstlane(*(LAS unsigned*)(l + LDS_CTL)) != 0;
            }
        }
    }

    const int orow = src * MSH + rtile * BM + wm * 64 + (lane & 7), ocol = ct * BN + wn * 64 + (lane & 8) * 2 + 4 * (lane >> 4);
    float* const optr = p.out + (size_t)orow * NL + ocol;
    auto store_rows = [&](int mi, float sc) {
#pragma unroll
        for (int np = 0; np < 2; ++np) {
            const v4f a = acc[mi][2 * np] * sc, bb = acc[mi][2 * np + 1] * sc;
            v4f x, y;
#pragma unroll
            for (int j = 0; j < 4; ++j) {
                const int aj = __float_as_int(a[j]), bj = __float_as_int(bb[j]);
                x[j] = __int_as_float(__builtin_amdgcn_update_dpp(aj, bj, 0x128, 0xf, 0xc, false));
                y[j] = __int_as_float(__builtin_amdgcn_update_dpp(bj, aj, 0x128, 0xf, 0x3, false));
            }
            float* q = optr + (size_t)mi * 16 * NL + np * 32;
            asm volatile("global_store_dwordx4 %0, %1, off sc1\n\ts_nop 1" :: "v"(q), "v"(x) : "memory");
            asm volatile("global_store_dwordx4 %0, %1, off sc1\n\ts_nop 1" :: "v"(q + 8 * NL), "v"(y) : "memory");
        }
    };

    if (ok) {
        STAMP(3);
        int slot = 0;
        v8i Af[4], Bf[4];
        auto read_frags = [&]() {
            const unsigned so = slot * ST;
#pragma unroll
            for (int ni = 0; ni < 4; ++ni) {
                LAS char* q = l + so + aB[ni];
                const v2i b0 = __builtin_amdgcn_ds_read_tr8_b64_v2i32((LAS v2i*)(q));
                const v2i b1 = __builtin_amdgcn_ds_read_tr8_b64_v2i32((LAS v2i*)(q + 1024));
                const v2i b2 = __builtin_amdgcn_ds_read_tr8_b64_v2i32((LAS v2i*)(q + 8192));
                const v2i b3 = __builtin_amdgcn_ds_read_tr8_b64_v2i32((LAS v2i*)(q + 9216));
                Bf[ni] = (v8i){b0.x, b0.y, b1.x, b1.y, b2.x, b2.y, b3.x, b3.y};
            }
#pragma unroll
            for (int mi = 0; mi < 4; ++mi) {
                const v4i a0 = *(LAS v4i*)(l + so + aA0 + mi * 2048);
                const v4i a1 = *(LAS v4i*)(l + so + aA1 + mi * 2048);
                Af[mi] = (v8i){a0.x, a0.y, a0.z, a0.w, a1.x, a1.y, a1.z, a1.w};
            }
        };
        auto mma_row = [&](int mi) {
#pragma unroll
            for (int ni = 0; ni < 4; ++ni)
                acc[mi][ni] = __builtin_amdgcn_mfma_scale_f32_16x16x128_f8f6f4(Bf[ni], Af[mi], acc[mi][ni], 0, 0, 0, 0, 0, 0);
        };
        for (int k = 0; k < NT - 2; ++k) {
            read_frags();
            const int s2 = slot == 0 ? 2 : slot - 1;
            const unsigned char* baseA = Abase + (size_t)(k + 2) * KS;
            const unsigned char* baseB = p.w + (size_t)(k + 2) * BK * NL;
            const unsigned la = lbase + s2 * ST + wid * 4096, lb = lbase + s2 * ST + A_ST + wid * 2048;
#pragma unroll
            for (int mi = 0; mi < 4; ++mi) {
#pragma unroll
                for (int ni = 0; ni < 4; ++ni) {
                    acc[mi][ni] = __builtin_amdgcn_mfma_scale_f32_16x16x128_f8f6f4(Bf[ni], Af[mi], acc[mi][ni], 0, 0, 0, 0, 0, 0);
                    const int i = mi * 4 + ni;
                    if ((i & 1) == 1 && i < 12) {
                        const int j = i >> 1;
                        __builtin_amdgcn_sched_barrier(0);
                        if (j < 4) dma16(la + j * 1024, voffA[j], baseA); else dma16(lb + (j - 4) * 1024, voffB[j - 4], baseB);
                        __builtin_amdgcn_sched_barrier(0);
                    }
                }
            }
            asm volatile("s_waitcnt lgkmcnt(0)" ::: "memory");
            waitv<6>();
            bar();
            slot = slot == 2 ? 0 : slot + 1;
        }
        unsigned tmov = __hip_atomic_load(tmo, __ATOMIC_RELAXED, __HIP_MEMORY_SCOPE_AGENT);
        read_frags();
#pragma unroll
        for (int mi = 0; mi < 4; ++mi) mma_row(mi);
        asm volatile("s_waitcnt lgkmcnt(0)" ::: "memory");
        waitv<0>();
        bar();
        slot = slot == 2 ? 0 : slot + 1;
        asm volatile("" : "+v"(tmov));
        const float sc = __builtin_amdgcn_readfirstlane(tmov) != call ? scale : __builtin_nanf("");
        read_frags();
        mma_row(0); mma_row(1);
        __builtin_amdgcn_sched_barrier(0);
        store_rows(0, sc);
        __builtin_amdgcn_sched_barrier(0);
        mma_row(2);
        __builtin_amdgcn_sched_barrier(0);
        store_rows(1, sc);
        __builtin_amdgcn_sched_barrier(0);
        mma_row(3);
        __builtin_amdgcn_sched_barrier(0);
        STAMP(8);
        store_rows(2, sc);
        store_rows(3, sc);
    } else {
        waitv<0>();
#pragma unroll
        for (int mi = 0; mi < 4; ++mi) store_rows(mi, __builtin_nanf(""));
    }
}
}

extern "C" void kernel_launch(int rank, int nranks, ncclComm_t comm, hipStream_t stream,
                              void* const* d_in, const int* in_sizes, int n_in,
                              void* d_out, int out_size, const int* in_dtypes, int out_dtype,
                              void* const* d_out_all, void* const* d_ws_all, size_t ws_bytes) {
    using namespace agg;
    (void)comm; (void)in_dtypes; (void)out_dtype; (void)d_out_all;
    if (nranks != NR || n_in != 4 || (unsigned)rank >= (unsigned)NR || ws_bytes < WS_NEED ||
        in_sizes[0] != MSH * KD || in_sizes[1] != KD * NL || out_size != NR * MSH * NL) {
        fprintf(stderr, "ag_gemm: unexpected problem shape\n");
        return;
    }
    hipError_t e = hipFuncSetAttribute((const void*)ag_gemm, hipFuncAttributeMaxDynamicSharedMemorySize, LDS_BYTES);
    if (e != hipSuccess) { fprintf(stderr, "ag_gemm: hipFuncSetAttribute failed: %s\n", hipGetErrorString(e)); return; }
    Params p;
    p.x = (const unsigned char*)d_in[0]; p.w = (const unsigned char*)d_in[1];
    p.sx = (const float*)d_in[2]; p.sw = (const float*)d_in[3]; p.out = (float*)d_out;
    for (int i = 0; i < NR; ++i) p.ws[i] = (char*)d_ws_all[(rank + i) & 7];
    p.rank = rank;
    static const unsigned PERM[NR] = {0x51473602u, 0x12570364u, 0x72536104u, 0x26413705u, 0x74261053u, 0x53164702u, 0x76135240u, 0x14370625u};
    p.perm = PERM[rank];
    hipLaunchKernelGGL(ag_gemm, dim3(256), dim3(512), LDS_BYTES, stream, p);
    e = hipGetLastError();
    if (e != hipSuccess) fprintf(stderr, "ag_gemm: launch failed: %s\n", hipGetErrorString(e));
}
```

```cpp
#include <hip/hip_runtime.h>
#include <rccl/rccl.h>
#include <cstdio>
#include <cstdlib>

#define HDELAY 8
namespace agg {
#define LAS __attribute__((address_space(3)))
#define GAS __attribute__((address_space(1)))
typedef int v8i __attribute__((ext_vector_type(8)));
typedef int v4i __attribute__((ext_vector_type(4)));
typedef int v2i __attribute__((ext_vector_type(2)));
typedef unsigned v4u __attribute__((ext_vector_type(4)));
typedef float v4f __attribute__((ext_vector_type(4)));

constexpr int NR = 8;
constexpr int MSH = 512, KD = 4096, NL = 2048;
constexpr int BM = 256, BN = 128, BK = 128, NT = KD / BK;
constexpr int A_ST = BM * BK, B_ST = BK * BN, ST = A_ST + B_ST, NSTAGE = 3;
constexpr int LDS_POLL = NSTAGE * ST;
constexpr int LDS_FLAGS = LDS_POLL + 1024;
constexpr int LDS_CTL = LDS_FLAGS + 1024;
constexpr int LDS_BYTES = LDS_CTL + 64;
constexpr size_t OFF_CNT = 0;
constexpr size_t OFF_TMO = 32768;
constexpr size_t OFF_FLAG = 65536;
constexpr size_t OFF_STG = 1u << 20;
constexpr size_t STG_BYTES = (size_t)MSH * KD;
constexpr size_t OFF_DIAG = 8u << 20;
constexpr size_t WS_NEED = 32u << 20;
constexpr unsigned long long DEADLINE_TICKS = 50000000ull;

struct Params {
    const unsigned char* x; const unsigned char* w; const float* sx; const float* sw; float* out;
    char* ws[NR];
    int rank; unsigned perm;
};

__device__ __forceinline__ unsigned long long rt() { return __builtin_amdgcn_s_memrealtime(); }
template <int N> __device__ __forceinline__ void waitv() { asm volatile("s_waitcnt vmcnt(%0)" :: "n"(N) : "memory"); }
__device__ __forceinline__ void dma16(unsigned ldsaddr, unsigned voff, const void* base) {
    asm volatile("s_mov_b32 m0, %0\n\ts_nop 0\n\tglobal_load_lds_dwordx4 %1, %2" :: "s"(ldsaddr), "v"(voff), "s"(base) : "memory");
}
__device__ __forceinline__ void dma4_sys(unsigned ldsaddr, unsigned voff, const void* base) {
    asm volatile("s_mov_b32 m0, %0\n\ts_nop 0\n\tglobal_load_lds_dword %1, %2 sc0 sc1" :: "s"(ldsaddr), "v"(voff), "s"(base) : "memory");
}
__device__ __forceinline__ void dma16_sys(unsigned ldsaddr, unsigned voff, const void* base) {
    asm volatile("s_mov_b32 m0, %0\n\ts_nop 0\n\tglobal_load_lds_dwordx4 %1, %2 sc0 sc1" :: "s"(ldsaddr), "v"(voff), "s"(base) : "memory");
}
__device__ __forceinline__ const unsigned char* uni(const void* p) {
    const unsigned long long v = (unsigned long long)p;
    const unsigned lo = __builtin_amdgcn_readfirstlane((unsigned)v), hi = __builtin_amdgcn_readfirstlane((unsigned)(v >> 32));
    return (const unsigned char*)(((unsigned long long)hi << 32) | lo);
}
__device__ __forceinline__ void bar() { asm volatile("" ::: "memory"); __builtin_amdgcn_s_barrier(); asm volatile("" ::: "memory"); }

#define STAMP(i) do {} while (0)

__global__ __launch_bounds__(512, 2) void ag_gemm(Params p) {
    extern __shared__ __attribute__((aligned(16))) char smem[];
    LAS char* l = (LAS char*)smem;
    const unsigned lbase = (unsigned)(size_t)l;
    const int tid = threadIdx.x, lane = tid & 63;
    const int wid = __builtin_amdgcn_readfirstlane(tid >> 6);
    const int b = blockIdx.x, g = b & 7, ti = b >> 3, rtile = ti & 1, ct = ti >> 1;
    const int q = (p.perm >> (4 * g)) & 7;
    const int src = (p.rank + q) & 7;
    STAMP(0);
    char* const self = p.ws[0];
    char* const w1 = p.ws[1]; char* const w2 = p.ws[2]; char* const w3 = p.ws[3]; char* const w4 = p.ws[4];
    char* const w5 = p.ws[5]; char* const w6 = p.ws[6]; char* const w7 = p.ws[7];
    asm volatile("" :: "s"(self), "s"(w1), "s"(w2), "s"(w3), "s"(w4), "s"(w5), "s"(w6), "s"(w7), "s"(p.x), "s"(p.w));
    typedef const __attribute__((address_space(4))) float* cfp;
    const float scale = *(cfp)p.sx * *(cfp)p.sw;

    unsigned voffB[2];
#pragma unroll
    for (int j = 0; j < 2; ++j) {
        const int pb = wid * 2 + j, k = pb * 8 + (lane >> 3);
        const int ch = (lane & 7) ^ (((lane >> 4) & 3) | (((pb >> 1) & 1) << 2));
        voffB[j] = (unsigned)(k * NL + ct * BN + ch * 16);
    }
    auto issueB = [&](int kt, int slot) {
        const unsigned char* base = p.w + (size_t)kt * BK * NL;
#pragma unroll
        for (int j = 0; j < 2; ++j) dma16(lbase + slot * ST + A_ST + (wid * 2 + j) * 1024, voffB[j], base);
    };

    if (wid < 4) {
        const int skt = b >> 3, sch = tid & 7, srow = (b & 7) * 64 + (tid >> 3);
        const unsigned char* sp = p.x + (size_t)srow * KD + skt * BK + sch * 16;
        const v4u sv0 = *(const GAS v4u*)sp, sv1 = *(const GAS v4u*)(sp + 32 * KD);
        const unsigned cnt = __hip_atomic_load((GAS unsigned*)(self + OFF_CNT + b * 128), __ATOMIC_RELAXED, __HIP_MEMORY_SCOPE_AGENT);
        const unsigned c = __builtin_amdgcn_readfirstlane(cnt) + 1;
        char* dst = self + OFF_STG + (c & 1) * STG_BYTES + (size_t)skt * (MSH * BK) + srow * BK + sch * 16;
        asm volatile("global_store_dwordx4 %0, %1, off sc0 sc1\n\tglobal_store_dwordx4 %2, %3, off sc0 sc1\n\ts_nop 1" :: "v"(dst), "v"(sv0), "v"(dst + 32 * BK), "v"(sv1) : "memory");
        if (tid == 0) { *(LAS unsigned*)(l + LDS_CTL + 4) = c; *(LAS unsigned*)(l + LDS_CTL + 8) = 0; }
        issueB(0, 0); issueB(1, 1);
        asm volatile("s_waitcnt vmcnt(4) lgkmcnt(0)" ::: "memory");
    } else {
        issueB(0, 0); issueB(1, 1);
    }
    bar();
    const unsigned call = __builtin_amdgcn_readfirstlane(*(LAS unsigned*)(l + LDS_CTL + 4));
    if (wid == 1 && lane == 0) {
        const unsigned fo = (unsigned)(OFF_FLAG + p.rank * 1024 + b * 4);
        char* const pw[7] = {w1, w2, w3, w4, w5, w6, w7};
#pragma unroll
        for (int q = 0; q < 7; ++q)
            __hip_atomic_store((GAS unsigned*)(pw[q] + fo), call, __ATOMIC_RELAXED, __HIP_MEMORY_SCOPE_SYSTEM);
        __hip_atomic_store((GAS unsigned*)(self + OFF_CNT + b * 128), call, __ATOMIC_RELAXED, __HIP_MEMORY_SCOPE_AGENT);
    }
    STAMP(1);

    const unsigned char* Abase; unsigned RS, KS;
    const bool remote = q != 0;
    if (!remote) { Abase = p.x; RS = KD; KS = BK; }
    else { Abase = (const unsigned char*)(p.ws[q] + OFF_STG + (call & 1) * STG_BYTES); RS = BK; KS = MSH * BK; }
    Abase = uni(Abase);
    unsigned voffA[4];
#pragma unroll
    for (int j = 0; j < 4; ++j) {
        const int pa = wid * 4 + j, row = pa * 8 + (lane >> 3);
        const int ch = (lane & 7) ^ ((pa & 1) * 4 + (lane >> 4));
        voffA[j] = (unsigned)(rtile * BM + row) * RS + ch * 16;
    }
    auto issueA = [&](int kt, int slot) {
        const unsigned char* base = Abase + (size_t)kt * KS;
#pragma unroll
        for (int j = 0; j < 4; ++j) dma16(lbase + slot * ST + (wid * 4 + j) * 1024, voffA[j], base);
    };
    GAS unsigned* tmo = (GAS unsigned*)(self + OFF_TMO);
    const unsigned char* fbase = uni(self + OFF_FLAG + src * 1024);
    auto gave_up = [&](unsigned long long t0) -> bool {
        if (__builtin_amdgcn_readfirstlane(__hip_atomic_load(tmo, __ATOMIC_RELAXED, __HIP_MEMORY_SCOPE_AGENT)) != call && rt() - t0 <= DEADLINE_TICKS) return false;
        if (lane == 0) __hip_atomic_store(tmo, call, __ATOMIC_RELAXED, __HIP_MEMORY_SCOPE_AGENT);
        return true;
    };
    bool ok = true;
    if (remote) {
        if (wid == 0) {
            const unsigned pv = (lane & 15) * 4;
            const unsigned long long t0 = rt();
            unsigned good = 1;
#pragma unroll
            for (int i = 0; i < 3; ++i) { dma4_sys(lbase + LDS_POLL + i * 256, pv, fbase); __builtin_amdgcn_s_sleep(5); }
            for (unsigned it = 0;; ++it) {
                dma4_sys(lbase + LDS_POLL + ((it + 3) & 3) * 256, pv, fbase);
                waitv<3>();
                const unsigned f = *(volatile LAS unsigned*)(l + LDS_POLL + (it & 3) * 256 + lane * 4);
                if (__all(f >= call)) break;
                if ((it & 63) == 63 && gave_up(t0)) { good = 0; break; }
            }
            if (lane == 0) *(LAS unsigned*)(l + LDS_CTL) = good;
            asm volatile("s_waitcnt lgkmcnt(0)" ::: "memory");
            STAMP(10);
        }
        bar();
        ok = __builtin_amdgcn_readfirstlane(*(LAS unsigned*)(l + LDS_CTL)) != 0;
    }
    STAMP(2);
    if (ok) {
        issueA(0, 0);
        if (remote && wid == 0) dma16_sys(lbase + LDS_FLAGS, lane * 16, fbase);
        issueA(1, 1);
    }

    const int wm = wid >> 1, wn = wid & 1;
    const unsigned sA = (lane & 15) >> 1, g4 = lane >> 4;
    const unsigned aA0 = (unsigned)(wm * 64 + (lane & 15)) * 128 + ((g4 ^ sA) * 16), aA1 = aA0 ^ 64;
    const unsigned rB = (lane & 15) >> 1, sB = ((rB >> 1) & 3) | ((g4 & 1) << 2);
    unsigned aB[4];
#pragma unroll
    for (int ni = 0; ni < 4; ++ni) aB[ni] = A_ST + (16 * g4 + rB) * 128 + (((wn * 4 + ni) ^ sB) * 16) + (lane & 1) * 8;

    v4f acc[4][4];
#pragma unroll
    for (int mi = 0; mi < 4; ++mi)
#pragma unroll
        for (int ni = 0; ni < 4; ++ni) acc[mi][ni] = (v4f){0.f, 0.f, 0.f, 0.f};

    if (ok) {
        waitv<4>();
        bar();
        if (remote) {
            const v4u f = *(LAS v4u*)(l + LDS_FLAGS + lane * 16);
            if (!__all(f.x >= call && f.y >= call && f.z >= call && f.w >= call)) {
                if (wid == 0) {
                    const char* fp = (const char*)fbase + lane * 16;
                    const unsigned long long t0 = rt();
                    unsigned good = 1;
                    for (unsigned it = 1;; ++it) {
                        v4u h;
                        asm volatile("global_load_dwordx4 %0, %1, off sc0 sc1\n\ts_waitcnt vmcnt(0)" : "=&v"(h) : "v"(fp) : "memory");
                        if (__all(h.x >= call && h.y >= call && h.z >= call && h.w >= call)) break;
                        if ((it & 15) == 0 && gave_up(t0)) { good = 0; break; }
                        __builtin_amdgcn_s_sleep(2);
                    }
                    if (lane == 0) *(LAS unsigned*)(l + LDS_CTL) = good;
                    asm volatile("s_waitcnt lgkmcnt(0)" ::: "memory");
                }
                bar();
                ok = __builtin_amdgcn_readfirstlane(*(LAS unsigned*)(l + LDS_CTL)) != 0;
            }
        }
    }

    const int orow = src * MSH + rtile * BM + wm * 64 + (lane & 7), ocol = ct * BN + wn * 64 + (lane & 8) * 2 + 4 * (lane >> 4);
    float* const optr = p.out + (size_t)orow * NL + ocol;
    auto store_rows = [&](int mi, float sc) {
#pragma unroll
        for (int np = 0; np < 2; ++np) {
            const v4f a = acc[mi][2 * np] * sc, bb = acc[mi][2 * np + 1] * sc;
            v4f x, y;
#pragma unroll
            for (int j = 0; j < 4; ++j) {
                const int aj = __float_as_int(a[j]), bj = __float_as_int(bb[j]);
                x[j] = __int_as_float(__builtin_amdgcn_update_dpp(aj, bj, 0x128, 0xf, 0xc, false));
                y[j] = __int_as_float(__builtin_amdgcn_update_dpp(bj, aj, 0x128, 0xf, 0x3, false));
            }
            float* q = optr + (size_t)mi * 16 * NL + np * 32;
            asm volatile("global_store_dwordx4 %0, %1, off sc1\n\ts_nop 1" :: "v"(q), "v"(x) : "memory");
            asm volatile("global_store_dwordx4 %0, %1, off sc1\n\ts_nop 1" :: "v"(q + 8 * NL), "v"(y) : "memory");
        }
    };

    if (ok) {
        STAMP(3);
        int slot = 0;
        v8i Af[4], Bf[4];
        auto read_frags = [&]() {
            const unsigned so = slot * ST;
#pragma unroll
            for (int ni = 0; ni < 4; ++ni) {
                LAS char* q = l + so + aB[ni];
                const v2i b0 = __builtin_amdgcn_ds_read_tr8_b64_v2i32((LAS v2i*)(q));
                const v2i b1 = __builtin_amdgcn_ds_read_tr8_b64_v2i32((LAS v2i*)(q + 1024));
                const v2i b2 = __builtin_amdgcn_ds_read_tr8_b64_v2i32((LAS v2i*)(q + 8192));
                const v2i b3 = __builtin_amdgcn_ds_read_tr8_b64_v2i32((LAS v2i*)(q + 9216));
                Bf[ni] = (v8i){b0.x, b0.y, b1.x, b1.y, b2.x, b2.y, b3.x, b3.y};
            }
#pragma unroll
            for (int mi = 0; mi < 4; ++mi) {
                const v4i a0 = *(LAS v4i*)(l + so + aA0 + mi * 2048);
                const v4i a1 = *(LAS v4i*)(l + so + aA1 + mi * 2048);
                Af[mi] = (v8i){a0.x, a0.y, a0.z, a0.w, a1.x, a1.y, a1.z, a1.w};
            }
        };
        auto mma_row = [&](int mi) {
#pragma unroll
            for (int ni = 0; ni < 4; ++ni)
                acc[mi][ni] = __builtin_amdgcn_mfma_scale_f32_16x16x128_f8f6f4(Bf[ni], Af[mi], acc[mi][ni], 0, 0, 0, 0, 0, 0);
        };
        auto next_slot = [&]() { slot = slot == 2 ? 0 : slot + 1; };
        auto pieceA = [&](int j, int kt, int sl) { dma16(lbase + sl * ST + (wid * 4 + j) * 1024, voffA[j], Abase + (size_t)kt * KS); };
        auto pieceB = [&](int j, int kt, int sl) { dma16(lbase + sl * ST + A_ST + (wid * 2 + j) * 1024, voffB[j], p.w + (size_t)kt * BK * NL); };
        auto tile = [&](auto&& issue) {
            read_frags();
#pragma unroll
            for (int mi = 0; mi < 4; ++mi) {
#pragma unroll
                for (int ni = 0; ni < 4; ++ni) {
                    acc[mi][ni] = __builtin_amdgcn_mfma_scale_f32_16x16x128_f8f6f4(Bf[ni], Af[mi], acc[mi][ni], 0, 0, 0, 0, 0, 0);
                    const int i = mi * 4 + ni;
                    if ((i & 1) == 1 && i < 12) {
                        __builtin_amdgcn_sched_barrier(0);
                        issue(i >> 1);
                        __builtin_amdgcn_sched_barrier(0);
                    }
                }
            }
            asm volatile("s_waitcnt lgkmcnt(0)" ::: "memory");
        };
        auto last_tile = [&](float sc) {
            read_frags();
            mma_row(0); mma_row(1);
            __builtin_amdgcn_sched_barrier(0);
            store_rows(0, sc);
            __builtin_amdgcn_sched_barrier(0);
            mma_row(2);
            __builtin_amdgcn_sched_barrier(0);
            store_rows(1, sc);
            __builtin_amdgcn_sched_barrier(0);
            mma_row(3);
            __builtin_amdgcn_sched_barrier(0);
            store_rows(2, sc);
            store_rows(3, sc);
        };
        for (int k = 0; k < NT - 5; ++k) {
            const int s2 = slot == 0 ? 2 : slot - 1;
            tile([&](int j) { if (j < 4) pieceA(j, k + 2, s2); else pieceB(j - 4, k + 2, s2); });
            waitv<6>();
            bar();
            next_slot();
        }
        static_assert(NT == 32, "");
        unsigned tmov = __hip_atomic_load(tmo, __ATOMIC_RELAXED, __HIP_MEMORY_SCOPE_AGENT);
        LAS unsigned* hb = (LAS unsigned*)(l + LDS_CTL + 8);
        if (wid < 4) {
            tile([&](int j) { if (j < 4) pieceA(j, 29, 2); else pieceB(j - 4, 29, 2); });
            waitv<6>(); bar(); next_slot();
            tile([&](int j) { if (j < 4) pieceA(j, 30, 0); else pieceB(j - 4, 30, 0); });
            waitv<6>(); bar(); next_slot();
            tile([&](int j) { if (j < 4) pieceA(j, 31, 1); else pieceB(j - 4, 31, 1); });
            waitv<6>(); bar(); next_slot();
            tile([&](int) {});
            waitv<0>(); bar(); next_slot();
            asm volatile("" : "+v"(tmov));
            const float sc = __builtin_amdgcn_readfirstlane(tmov) != call ? scale : __builtin_nanf("");
            STAMP(8);
            last_tile(sc);
        } else {
            tile([&](int j) { if (j >= 4) pieceB(j - 4, 29, 2); });
            waitv<2>(); bar(); next_slot();
            tile([&](int j) { if (j < 4) pieceA(j, 29, 2); else pieceB(j - 4, 30, 0); });
            waitv<6>(); bar();
            pieceB(0, 31, 1); pieceB(1, 31, 1);
            __builtin_amdgcn_s_sleep(HDELAY);
#pragma unroll
            for (int j = 0; j < 4; ++j) pieceA(j, 30, 0);
            waitv<6>(); bar();
            slot = 2;
            tile([&](int j) { if (j < 4) pieceA(j, 31, 1); });
            waitv<8>(); bar();
            auto half_bar = [&](unsigned target) {
                if (lane == 0) __hip_atomic_fetch_add(hb, 1u, __ATOMIC_RELAXED, __HIP_MEMORY_SCOPE_WORKGROUP);
                while (__builtin_amdgcn_readfirstlane(__hip_atomic_load(hb, __ATOMIC_RELAXED, __HIP_MEMORY_SCOPE_WORKGROUP)) < target) __builtin_amdgcn_s_sleep(0);
                asm volatile("" ::: "memory");
            };
            waitv<4>(); half_bar(4);
            slot = 0;
            tile([&](int) {});
            waitv<0>(); half_bar(8);
            slot = 1;
            asm volatile("" : "+v"(tmov));
            const float sc = __builtin_amdgcn_readfirstlane(tmov) != call ? scale : __builtin_nanf("");
            last_tile(sc);
        }
    } else {
        waitv<0>();
#pragma unroll
        for (int mi = 0; mi < 4; ++mi) store_rows(mi, __builtin_nanf(""));
    }
}
}

extern "C" void kernel_launch(int rank, int nranks, ncclComm_t comm, hipStream_t stream,
                              void* const* d_in, const int* in_sizes, int n_in,
                              void* d_out, int out_size, const int* in_dtypes, int out_dtype,
                              void* const* d_out_all, void* const* d_ws_all, size_t ws_bytes) {
    using namespace agg;
    (void)comm; (void)in_dtypes; (void)out_dtype; (void)d_out_all;
    if (nranks != NR || n_in != 4 || (unsigned)rank >= (unsigned)NR || ws_bytes < WS_NEED ||
        in_sizes[0] != MSH * KD || in_sizes[1] != KD * NL || out_size != NR * MSH * NL) {
        fprintf(stderr, "ag_gemm: unexpected problem shape\n");
        return;
    }
    hipError_t e = hipFuncSetAttribute((const void*)ag_gemm, hipFuncAttributeMaxDynamicSharedMemorySize, LDS_BYTES);
    if (e != hipSuccess) { fprintf(stderr, "ag_gemm: hipFuncSetAttribute failed: %s\n", hipGetErrorString(e)); return; }
    Params p;
    p.x = (const unsigned char*)d_in[0]; p.w = (const unsigned char*)d_in[1];
    p.sx = (const float*)d_in[2]; p.sw = (const float*)d_in[3]; p.out = (float*)d_out;
    for (int i = 0; i < NR; ++i) p.ws[i] = (char*)d_ws_all[(rank + i) & 7];
    p.rank = rank;
    static const unsigned PERM[NR] = {0x51473602u, 0x12570364u, 0x72536104u, 0x26413705u, 0x74261053u, 0x53164702u, 0x76135240u, 0x14370625u};
    p.perm = PERM[rank];
    hipLaunchKernelGGL(ag_gemm, dim3(256), dim3(512), LDS_BYTES, stream, p);
    e = hipGetLastError();
    if (e != hipSuccess) fprintf(stderr, "ag_gemm: launch failed: %s\n", hipGetErrorString(e));
}
```

```cpp
#include <hip/hip_runtime.h>
#include <rccl/rccl.h>
#include <cstdio>

namespace agg {
#define LAS __attribute__((address_space(3)))
#define GAS __attribute__((address_space(1)))
typedef int v8i __attribute__((ext_vector_type(8)));
typedef int v4i __attribute__((ext_vector_type(4)));
typedef int v2i __attribute__((ext_vector_type(2)));
typedef unsigned v4u __attribute__((ext_vector_type(4)));
typedef float v4f __attribute__((ext_vector_type(4)));

constexpr int NR = 8;
constexpr int MSH = 512, KD = 4096, NL = 2048;
constexpr int BM = 256, BN = 128, BK = 128, NT = KD / BK;
constexpr int A_ST = BM * BK, B_ST = BK * BN, ST = A_ST + B_ST, NSTAGE = 3;
constexpr int LDS_POLL = NSTAGE * ST;
constexpr int LDS_FLAGS = LDS_POLL + 1024;
constexpr int LDS_CTL = LDS_FLAGS + 1024;
constexpr int LDS_BYTES = LDS_CTL + 64;
constexpr size_t OFF_CNT = 0;
constexpr size_t OFF_TMO = 32768;
constexpr size_t OFF_FLAG = 65536;
constexpr size_t OFF_STG = 1u << 20;
constexpr size_t STG_BYTES = (size_t)MSH * KD;
constexpr size_t WS_NEED = OFF_STG + 2 * STG_BYTES;
constexpr unsigned long long DEADLINE_TICKS = 50000000ull;

struct Params {
    const unsigned char* x; const unsigned char* w; const float* sx; const float* sw; float* out;
    char* ws[NR];
    int rank; unsigned perm;
};

__device__ __forceinline__ unsigned long long rt() { return __builtin_amdgcn_s_memrealtime(); }
template <int N> __device__ __forceinline__ void waitv() { asm volatile("s_waitcnt vmcnt(%0)" :: "n"(N) : "memory"); }
__device__ __forceinline__ void dma16(unsigned ldsaddr, unsigned voff, const void* base) {
    asm volatile("s_mov_b32 m0, %0\n\ts_nop 0\n\tglobal_load_lds_dwordx4 %1, %2" :: "s"(ldsaddr), "v"(voff), "s"(base) : "memory");
}
__device__ __forceinline__ void dma4_sys(unsigned ldsaddr, unsigned voff, const void* base) {
    asm volatile("s_mov_b32 m0, %0\n\ts_nop 0\n\tglobal_load_lds_dword %1, %2 sc0 sc1" :: "s"(ldsaddr), "v"(voff), "s"(base) : "memory");
}
__device__ __forceinline__ void dma16_sys(unsigned ldsaddr, unsigned voff, const void* base) {
    asm volatile("s_mov_b32 m0, %0\n\ts_nop 0\n\tglobal_load_lds_dwordx4 %1, %2 sc0 sc1" :: "s"(ldsaddr), "v"(voff), "s"(base) : "memory");
}
__device__ __forceinline__ const unsigned char* uni(const void* p) {
    const unsigned long long v = (unsigned long long)p;
    const unsigned lo = __builtin_amdgcn_readfirstlane((unsigned)v), hi = __builtin_amdgcn_readfirstlane((unsigned)(v >> 32));
    return (const unsigned char*)(((unsigned long long)hi << 32) | lo);
}
__device__ __forceinline__ void bar() { asm volatile("" ::: "memory"); __builtin_amdgcn_s_barrier(); asm volatile("" ::: "memory"); }


__global__ __launch_bounds__(512, 2) void ag_gemm(Params p) {
    extern __shared__ __attribute__((aligned(16))) char smem[];
    LAS char* l = (LAS char*)smem;
    const unsigned lbase = (unsigned)(size_t)l;
    const int tid = threadIdx.x, lane = tid & 63;
    const int wid = __builtin_amdgcn_readfirstlane(tid >> 6);
    const int b = blockIdx.x, g = b & 7, ti = b >> 3, rtile = ti & 1, ct = ti >> 1;
    const int q = (p.perm >> (4 * g)) & 7;
    const int src = (p.rank + q) & 7;
    char* const self = p.ws[0];
    char* const w1 = p.ws[1]; char* const w2 = p.ws[2]; char* const w3 = p.ws[3]; char* const w4 = p.ws[4];
    char* const w5 = p.ws[5]; char* const w6 = p.ws[6]; char* const w7 = p.ws[7];
    asm volatile("" :: "s"(self), "s"(w1), "s"(w2), "s"(w3), "s"(w4), "s"(w5), "s"(w6), "s"(w7), "s"(p.x), "s"(p.w));
    typedef const __attribute__((address_space(4))) float* cfp;
    const float scale = *(cfp)p.sx * *(cfp)p.sw;

    unsigned voffB[2];
#pragma unroll
    for (int j = 0; j < 2; ++j) {
        const int pb = wid * 2 + j, k = pb * 8 + (lane >> 3);
        const int ch = (lane & 7) ^ (((lane >> 4) & 3) | (((pb >> 1) & 1) << 2));
        voffB[j] = (unsigned)(k * NL + ct * BN + ch * 16);
    }
    auto issueB = [&](int kt, int slot) {
        const unsigned char* base = p.w + (size_t)kt * BK * NL;
#pragma unroll
        for (int j = 0; j < 2; ++j) dma16(lbase + slot * ST + A_ST + (wid * 2 + j) * 1024, voffB[j], base);
    };

    if (wid < 4) {
        const int skt = b >> 3, sch = tid & 7, srow = (b & 7) * 64 + (tid >> 3);
        const unsigned char* sp = p.x + (size_t)srow * KD + skt * BK + sch * 16;
        const v4u sv0 = *(const GAS v4u*)sp, sv1 = *(const GAS v4u*)(sp + 32 * KD);
        const unsigned cnt = __hip_atomic_load((GAS unsigned*)(self + OFF_CNT + b * 128), __ATOMIC_RELAXED, __HIP_MEMORY_SCOPE_AGENT);
        const unsigned c = __builtin_amdgcn_readfirstlane(cnt) + 1;
        char* dst = self + OFF_STG + (c & 1) * STG_BYTES + (size_t)skt * (MSH * BK) + srow * BK + sch * 16;
        asm volatile("global_store_dwordx4 %0, %1, off sc0 sc1\n\tglobal_store_dwordx4 %2, %3, off sc0 sc1\n\ts_nop 1" :: "v"(dst), "v"(sv0), "v"(dst + 32 * BK), "v"(sv1) : "memory");
        if (tid == 0) { *(LAS unsigned*)(l + LDS_CTL + 4) = c; *(LAS unsigned*)(l + LDS_CTL + 8) = 0; }
        issueB(0, 0); issueB(1, 1);
        asm volatile("s_waitcnt vmcnt(4) lgkmcnt(0)" ::: "memory");
    } else {
        issueB(0, 0); issueB(1, 1);
    }
    bar();
    const unsigned call = __builtin_amdgcn_readfirstlane(*(LAS unsigned*)(l + LDS_CTL + 4));
    if (wid == 1 && lane == 0) {
        const unsigned fo = (unsigned)(OFF_FLAG + p.rank * 1024 + b * 4);
        char* const pw[7] = {w1, w2, w3, w4, w5, w6, w7};
#pragma unroll
        for (int q = 0; q < 7; ++q)
            __hip_atomic_store((GAS unsigned*)(pw[q] + fo), call, __ATOMIC_RELAXED, __HIP_MEMORY_SCOPE_SYSTEM);
        __hip_atomic_store((GAS unsigned*)(self + OFF_CNT + b * 128), call, __ATOMIC_RELAXED, __HIP_MEMORY_SCOPE_AGENT);
    }

    const unsigned char* Abase; unsigned RS, KS;
    const bool remote = q != 0;
    if (!remote) { Abase = p.x; RS = KD; KS = BK; }
    else { Abase = (const unsigned char*)(p.ws[q] + OFF_STG + (call & 1) * STG_BYTES); RS = BK; KS = MSH * BK; }
    Abase = uni(Abase);
    unsigned voffA[4];
#pragma unroll
    for (int j = 0; j < 4; ++j) {
        const int pa = wid * 4 + j, row = pa * 8 + (lane >> 3);
        const int ch = (lane & 7) ^ ((pa & 1) * 4 + (lane >> 4));
        voffA[j] = (unsigned)(rtile * BM + row) * RS + ch * 16;
    }
    auto issueA = [&](int kt, int slot) {
        const unsigned char* base = Abase + (size_t)kt * KS;
#pragma unroll
        for (int j = 0; j < 4; ++j) dma16(lbase + slot * ST + (wid * 4 + j) * 1024, voffA[j], base);
    };
    GAS unsigned* tmo = (GAS unsigned*)(self + OFF_TMO);
    const unsigned char* fbase = uni(self + OFF_FLAG + src * 1024);
    auto gave_up = [&](unsigned long long t0) -> bool {
        if (__builtin_amdgcn_readfirstlane(__hip_atomic_load(tmo, __ATOMIC_RELAXED, __HIP_MEMORY_SCOPE_AGENT)) != call && rt() - t0 <= DEADLINE_TICKS) return false;
        if (lane == 0) __hip_atomic_store(tmo, call, __ATOMIC_RELAXED, __HIP_MEMORY_SCOPE_AGENT);
        return true;
    };
    bool ok = true;
    if (remote) {
        if (wid == 0) {
            const unsigned pv = (lane & 15) * 4;
            const unsigned long long t0 = rt();
            unsigned good = 1;
#pragma unroll
            for (int i = 0; i < 3; ++i) { dma4_sys(lbase + LDS_POLL + i * 256, pv, fbase); __builtin_amdgcn_s_sleep(5); }
            for (unsigned it = 0;; ++it) {
                dma4_sys(lbase + LDS_POLL + ((it + 3) & 3) * 256, pv, fbase);
                waitv<3>();
                const unsigned f = *(volatile LAS unsigned*)(l + LDS_POLL + (it & 3) * 256 + lane * 4);
                if (__all(f >= call)) break;
                if ((it & 63) == 63 && gave_up(t0)) { good = 0; break; }
            }
            if (lane == 0) *(LAS unsigned*)(l + LDS_CTL) = good;
            asm volatile("s_waitcnt lgkmcnt(0)" ::: "memory");
        }
        bar();
        ok = __builtin_amdgcn_readfirstlane(*(LAS unsigned*)(l + LDS_CTL)) != 0;
    }
    if (ok) {
        issueA(0, 0);
        if (remote && wid == 0) dma16_sys(lbase + LDS_FLAGS, lane * 16, fbase);
        issueA(1, 1);
    }

    const int wm = wid >> 1, wn = wid & 1;
    const unsigned sA = (lane & 15) >> 1, g4 = lane >> 4;
    const unsigned aA0 = (unsigned)(wm * 64 + (lane & 15)) * 128 + ((g4 ^ sA) * 16), aA1 = aA0 ^ 64;
    const unsigned rB = (lane & 15) >> 1, sB = ((rB >> 1) & 3) | ((g4 & 1) << 2);
    unsigned aB[4];
#pragma unroll
    for (int ni = 0; ni < 4; ++ni) aB[ni] = A_ST + (16 * g4 + rB) * 128 + (((wn * 4 + ni) ^ sB) * 16) + (lane & 1) * 8;

    v4f acc[4][4];
#pragma unroll
    for (int mi = 0; mi < 4; ++mi)
#pragma unroll
        for (int ni = 0; ni < 4; ++ni) acc[mi][ni] = (v4f){0.f, 0.f, 0.f, 0.f};

    if (ok) {
        waitv<4>();
        bar();
        if (remote) {
            const v4u f = *(LAS v4u*)(l + LDS_FLAGS + lane * 16);
            if (!__all(f.x >= call && f.y >= call && f.z >= call && f.w >= call)) {
                if (wid == 0) {
                    const char* fp = (const char*)fbase + lane * 16;
                    const unsigned long long t0 = rt();
                    unsigned good = 1;
                    for (unsigned it = 1;; ++it) {
                        v4u h;
                        asm volatile("global_load_dwordx4 %0, %1, off sc0 sc1\n\ts_waitcnt vmcnt(0)" : "=&v"(h) : "v"(fp) : "memory");
                        if (__all(h.x >= call && h.y >= call && h.z >= call && h.w >= call)) break;
                        if ((it & 15) == 0 && gave_up(t0)) { good = 0; break; }
                        __builtin_amdgcn_s_sleep(2);
                    }
                    if (lane == 0) *(LAS unsigned*)(l + LDS_CTL) = good;
                    asm volatile("s_waitcnt lgkmcnt(0)" ::: "memory");
                }
                bar();
                ok = __builtin_amdgcn_readfirstlane(*(LAS unsigned*)(l + LDS_CTL)) != 0;
            }
        }
    }

    const int orow = src * MSH + rtile * BM + wm * 64 + (lane & 7), ocol = ct * BN + wn * 64 + (lane & 8) * 2 + 4 * (lane >> 4);
    float* const optr = p.out + (size_t)orow * NL + ocol;
    auto store_rows = [&](int mi, float sc) {
#pragma unroll
        for (int np = 0; np < 2; ++np) {
            const v4f a = acc[mi][2 * np] * sc, bb = acc[mi][2 * np + 1] * sc;
            v4f x, y;
#pragma unroll
            for (int j = 0; j < 4; ++j) {
                const int aj = __float_as_int(a[j]), bj = __float_as_int(bb[j]);
                x[j] = __int_as_float(__builtin_amdgcn_update_dpp(aj, bj, 0x128, 0xf, 0xc, false));
                y[j] = __int_as_float(__builtin_amdgcn_update_dpp(bj, aj, 0x128, 0xf, 0x3, false));
            }
            float* q = optr + (size_t)mi * 16 * NL + np * 32;
            asm volatile("global_store_dwordx4 %0, %1, off sc1\n\ts_nop 1" :: "v"(q), "v"(x) : "memory");
            asm volatile("global_store_dwordx4 %0, %1, off sc1\n\ts_nop 1" :: "v"(q + 8 * NL), "v"(y) : "memory");
        }
    };

    if (ok) {
        int slot = 0;
        v8i Af[4], Bf[4];
        auto read_frags = [&]() {
            const unsigned so = slot * ST;
#pragma unroll
            for (int ni = 0; ni < 4; ++ni) {
                LAS char* q = l + so + aB[ni];
                const v2i b0 = __builtin_amdgcn_ds_read_tr8_b64_v2i32((LAS v2i*)(q));
                const v2i b1 = __builtin_amdgcn_ds_read_tr8_b64_v2i32((LAS v2i*)(q + 1024));
                const v2i b2 = __builtin_amdgcn_ds_read_tr8_b64_v2i32((LAS v2i*)(q + 8192));
                const v2i b3 = __builtin_amdgcn_ds_read_tr8_b64_v2i32((LAS v2i*)(q + 9216));
                Bf[ni] = (v8i){b0.x, b0.y, b1.x, b1.y, b2.x, b2.y, b3.x, b3.y};
            }
#pragma unroll
            for (int mi = 0; mi < 4; ++mi) {
                const v4i a0 = *(LAS v4i*)(l + so + aA0 + mi * 2048);
                const v4i a1 = *(LAS v4i*)(l + so + aA1 + mi * 2048);
                Af[mi] = (v8i){a0.x, a0.y, a0.z, a0.w, a1.x, a1.y, a1.z, a1.w};
            }
        };
        auto mma_row = [&](int mi) {
#pragma unroll
            for (int ni = 0; ni < 4; ++ni)
                acc[mi][ni] = __builtin_amdgcn_mfma_scale_f32_16x16x128_f8f6f4(Bf[ni], Af[mi], acc[mi][ni], 0, 0, 0, 0, 0, 0);
        };
        auto next_slot = [&]() { slot = slot == 2 ? 0 : slot + 1; };
        auto pieceA = [&](int j, int kt, int sl) { dma16(lbase + sl * ST + (wid * 4 + j) * 1024, voffA[j], Abase + (size_t)kt * KS); };
        auto pieceB = [&](int j, int kt, int sl) { dma16(lbase + sl * ST + A_ST + (wid * 2 + j) * 1024, voffB[j], p.w + (size_t)kt * BK * NL); };
        auto tile = [&](auto&& issue) {
            read_frags();
#pragma unroll
            for (int mi = 0; mi < 4; ++mi) {
#pragma unroll
                for (int ni = 0; ni < 4; ++ni) {
                    acc[mi][ni] = __builtin_amdgcn_mfma_scale_f32_16x16x128_f8f6f4(Bf[ni], Af[mi], acc[mi][ni], 0, 0, 0, 0, 0, 0);
                    const int i = mi * 4 + ni;
                    if ((i & 1) == 1 && i < 12) {
                        __builtin_amdgcn_sched_barrier(0);
                        issue(i >> 1);
                        __builtin_amdgcn_sched_barrier(0);
                    }
                }
            }
            asm volatile("s_waitcnt lgkmcnt(0)" ::: "memory");
        };
        auto last_tile = [&](float sc) {
            read_frags();
            mma_row(0); mma_row(1);
            __builtin_amdgcn_sched_barrier(0);
            store_rows(0, sc);
            __builtin_amdgcn_sched_barrier(0);
            mma_row(2);
            __builtin_amdgcn_sched_barrier(0);
            store_rows(1, sc);
            __builtin_amdgcn_sched_barrier(0);
            mma_row(3);
            __builtin_amdgcn_sched_barrier(0);
            store_rows(2, sc);
            store_rows(3, sc);
        };
        for (int k = 0; k < NT - 5; ++k) {
            const int s2 = slot == 0 ? 2 : slot - 1;
            tile([&](int j) { if (j < 4) pieceA(j, k + 2, s2); else pieceB(j - 4, k + 2, s2); });
            waitv<6>();
            bar();
            next_slot();
        }
        static_assert(NT == 32, "");
        unsigned tmov = __hip_atomic_load(tmo, __ATOMIC_RELAXED, __HIP_MEMORY_SCOPE_AGENT);
        LAS unsigned* hb = (LAS unsigned*)(l + LDS_CTL + 8);
        if (wid < 4) {
            tile([&](int j) { if (j < 4) pieceA(j, 29, 2); else pieceB(j - 4, 29, 2); });
            waitv<6>(); bar(); next_slot();
            tile([&](int j) { if (j < 4) pieceA(j, 30, 0); else pieceB(j - 4, 30, 0); });
            waitv<6>(); bar(); next_slot();
            tile([&](int j) { if (j < 4) pieceA(j, 31, 1); else pieceB(j - 4, 31, 1); });
            waitv<6>(); bar(); next_slot();
            tile([&](int) {});
            waitv<0>(); bar(); next_slot();
            asm volatile("" : "+v"(tmov));
            const float sc = __builtin_amdgcn_readfirstlane(tmov) != call ? scale : __builtin_nanf("");
            last_tile(sc);
        } else {
            tile([&](int j) { if (j >= 4) pieceB(j - 4, 29, 2); });
            waitv<2>(); bar(); next_slot();
            tile([&](int j) { if (j < 4) pieceA(j, 29, 2); else pieceB(j - 4, 30, 0); });
            waitv<6>(); bar();
            pieceB(0, 31, 1); pieceB(1, 31, 1);
            __builtin_amdgcn_s_sleep(6);
#pragma unroll
            for (int j = 0; j < 4; ++j) pieceA(j, 30, 0);
            waitv<6>(); bar();
            slot = 2;
            tile([&](int j) { if (j < 4) pieceA(j, 31, 1); });
            waitv<8>(); bar();
            auto half_bar = [&](unsigned target) {
                if (lane == 0) __hip_atomic_fetch_add(hb, 1u, __ATOMIC_RELAXED, __HIP_MEMORY_SCOPE_WORKGROUP);
                while (__builtin_amdgcn_readfirstlane(__hip_atomic_load(hb, __ATOMIC_RELAXED, __HIP_MEMORY_SCOPE_WORKGROUP)) < target) __builtin_amdgcn_s_sleep(0);
                asm volatile("" ::: "memory");
            };
            waitv<4>(); half_bar(4);
            slot = 0;
            tile([&](int) {});
            waitv<0>(); half_bar(8);
            slot = 1;
            asm volatile("" : "+v"(tmov));
            const float sc = __builtin_amdgcn_readfirstlane(tmov) != call ? scale : __builtin_nanf("");
            last_tile(sc);
        }
    } else {
        waitv<0>();
#pragma unroll
        for (int mi = 0; mi < 4; ++mi) store_rows(mi, __builtin_nanf(""));
    }
}
}

extern "C" void kernel_launch(int rank, int nranks, ncclComm_t comm, hipStream_t stream,
                              void* const* d_in, const int* in_sizes, int n_in,
                              void* d_out, int out_size, const int* in_dtypes, int out_dtype,
                              void* const* d_out_all, void* const* d_ws_all, size_t ws_bytes) {
    using namespace agg;
    (void)comm; (void)in_dtypes; (void)out_dtype; (void)d_out_all;
    if (nranks != NR || n_in != 4 || (unsigned)rank >= (unsigned)NR || ws_bytes < WS_NEED ||
        in_sizes[0] != MSH * KD || in_sizes[1] != KD * NL || out_size != NR * MSH * NL) {
        fprintf(stderr, "ag_gemm: unexpected problem shape\n");
        return;
    }
    hipError_t e = hipFuncSetAttribute((const void*)ag_gemm, hipFuncAttributeMaxDynamicSharedMemorySize, LDS_BYTES);
    if (e != hipSuccess) { fprintf(stderr, "ag_gemm: hipFuncSetAttribute failed: %s\n", hipGetErrorString(e)); return; }
    Params p;
    p.x = (const unsigned char*)d_in[0]; p.w = (const unsigned char*)d_in[1];
    p.sx = (const float*)d_in[2]; p.sw = (const float*)d_in[3]; p.out = (float*)d_out;
    for (int i = 0; i < NR; ++i) p.ws[i] = (char*)d_ws_all[(rank + i) & 7];
    p.rank = rank;
    static const unsigned PERM[NR] = {0x51473602u, 0x12570364u, 0x72536104u, 0x26413705u, 0x74261053u, 0x53164702u, 0x76135240u, 0x14370625u};
    p.perm = PERM[rank];
    hipLaunchKernelGGL(ag_gemm, dim3(256), dim3(512), LDS_BYTES, stream, p);
    e = hipGetLastError();
    if (e != hipSuccess) fprintf(stderr, "ag_gemm: launch failed: %s\n", hipGetErrorString(e));
}
```

```cpp
#include <hip/hip_runtime.h>
#include <rccl/rccl.h>
#include <cstdio>

namespace agg {
#define LAS __attribute__((address_space(3)))
#define GAS __attribute__((address_space(1)))
typedef int v8i __attribute__((ext_vector_type(8)));
typedef int v4i __attribute__((ext_vector_type(4)));
typedef int v2i __attribute__((ext_vector_type(2)));
typedef unsigned v4u __attribute__((ext_vector_type(4)));
typedef float v4f __attribute__((ext_vector_type(4)));

constexpr int NR = 8;
constexpr int MSH = 512, KD = 4096, NL = 2048;
constexpr int BM = 256, BN = 128, BK = 128, NT = KD / BK;
constexpr int A_ST = BM * BK, B_ST = BK * BN, ST = A_ST + B_ST, NSTAGE = 3;
constexpr int LDS_POLL = NSTAGE * ST;
constexpr int LDS_FLAGS = LDS_POLL + 1024;
constexpr int LDS_CTL = LDS_FLAGS + 1024;
constexpr int LDS_BYTES = LDS_CTL + 64;
constexpr size_t OFF_CNT = 0;
constexpr size_t OFF_TMO = 32768;
constexpr size_t OFF_CLAIM = 49152;
constexpr size_t OFF_FLAG = 65536;
constexpr size_t OFF_STG = 1u << 20;
constexpr size_t STG_BYTES = (size_t)MSH * KD;
constexpr size_t WS_NEED = OFF_STG + 2 * STG_BYTES;
constexpr unsigned long long DEADLINE_TICKS = 50000000ull;

struct Params {
    const unsigned char* x; const unsigned char* w; const float* sx; const float* sw; float* out;
    char* ws[NR];
    int rank; unsigned perm;
};

__device__ __forceinline__ unsigned long long rt() { return __builtin_amdgcn_s_memrealtime(); }
template <int N> __device__ __forceinline__ void waitv() { asm volatile("s_waitcnt vmcnt(%0)" :: "n"(N) : "memory"); }
__device__ __forceinline__ void dma16(unsigned ldsaddr, unsigned voff, const void* base) {
    asm volatile("s_mov_b32 m0, %0\n\ts_nop 0\n\tglobal_load_lds_dwordx4 %1, %2" :: "s"(ldsaddr), "v"(voff), "s"(base) : "memory");
}
__device__ __forceinline__ void dma4_sys(unsigned ldsaddr, unsigned voff, const void* base) {
    asm volatile("s_mov_b32 m0, %0\n\ts_nop 0\n\tglobal_load_lds_dword %1, %2 sc0 sc1" :: "s"(ldsaddr), "v"(voff), "s"(base) : "memory");
}
__device__ __forceinline__ void dma16_sys(unsigned ldsaddr, unsigned voff, const void* base) {
    asm volatile("s_mov_b32 m0, %0\n\ts_nop 0\n\tglobal_load_lds_dwordx4 %1, %2 sc0 sc1" :: "s"(ldsaddr), "v"(voff), "s"(base) : "memory");
}
__device__ __forceinline__ const unsigned char* uni(const void* p) {
    const unsigned long long v = (unsigned long long)p;
    const unsigned lo = __builtin_amdgcn_readfirstlane((unsigned)v), hi = __builtin_amdgcn_readfirstlane((unsigned)(v >> 32));
    return (const unsigned char*)(((unsigned long long)hi << 32) | lo);
}
__device__ __forceinline__ void bar() { asm volatile("" ::: "memory"); __builtin_amdgcn_s_barrier(); asm volatile("" ::: "memory"); }


__global__ __launch_bounds__(512, 2) void ag_gemm(Params p) {
    extern __shared__ __attribute__((aligned(16))) char smem[];
    LAS char* l = (LAS char*)smem;
    const unsigned lbase = (unsigned)(size_t)l;
    const int tid = threadIdx.x, lane = tid & 63;
    const int wid = __builtin_amdgcn_readfirstlane(tid >> 6);
    const int b = blockIdx.x, ti = b >> 3, rtile = ti & 1, ct = ti >> 1;
    char* const self = p.ws[0];
    char* const w1 = p.ws[1]; char* const w2 = p.ws[2]; char* const w3 = p.ws[3]; char* const w4 = p.ws[4];
    char* const w5 = p.ws[5]; char* const w6 = p.ws[6]; char* const w7 = p.ws[7];
    asm volatile("" :: "s"(self), "s"(w1), "s"(w2), "s"(w3), "s"(w4), "s"(w5), "s"(w6), "s"(w7), "s"(p.x), "s"(p.w));
    typedef const __attribute__((address_space(4))) float* cfp;
    const float scale = *(cfp)p.sx * *(cfp)p.sw;

    unsigned voffB[2];
#pragma unroll
    for (int j = 0; j < 2; ++j) {
        const int pb = wid * 2 + j, k = pb * 8 + (lane >> 3);
        const int ch = (lane & 7) ^ (((lane >> 4) & 3) | (((pb >> 1) & 1) << 2));
        voffB[j] = (unsigned)(k * NL + ct * BN + ch * 16);
    }
    auto issueB = [&](int kt, int slot) {
        const unsigned char* base = p.w + (size_t)kt * BK * NL;
#pragma unroll
        for (int j = 0; j < 2; ++j) dma16(lbase + slot * ST + A_ST + (wid * 2 + j) * 1024, voffB[j], base);
    };

    if (wid < 4) {
        const int skt = b >> 3, sch = tid & 7, srow = (b & 7) * 64 + (tid >> 3);
        const unsigned char* sp = p.x + (size_t)srow * KD + skt * BK + sch * 16;
        const v4u sv0 = *(const GAS v4u*)sp, sv1 = *(const GAS v4u*)(sp + 32 * KD);
        const unsigned cnt = __hip_atomic_load((GAS unsigned*)(self + OFF_CNT + b * 128), __ATOMIC_RELAXED, __HIP_MEMORY_SCOPE_AGENT);
        const unsigned c = __builtin_amdgcn_readfirstlane(cnt) + 1;
        char* dst = self + OFF_STG + (c & 1) * STG_BYTES + (size_t)skt * (MSH * BK) + srow * BK + sch * 16;
        asm volatile("global_store_dwordx4 %0, %1, off sc0 sc1\n\tglobal_store_dwordx4 %2, %3, off sc0 sc1\n\ts_nop 1" :: "v"(dst), "v"(sv0), "v"(dst + 32 * BK), "v"(sv1) : "memory");
        if (wid == 0) {
            const unsigned xcc = (unsigned)__builtin_amdgcn_s_getreg(20 | (2 << 11)) & 7u;
            unsigned want = (p.perm >> (4 * xcc)) & 7u;
            GAS unsigned* mask = (GAS unsigned*)(self + OFF_CLAIM + ((c & 1) * 32 + ti) * 128);
            unsigned old = 0;
            if (lane == 0) asm volatile("global_atomic_or %0, %1, %2, off sc0" : "+v"(old) : "v"(mask), "v"(1u << want) : "memory");
            issueB(0, 0); issueB(1, 1);
            asm volatile("s_waitcnt vmcnt(4)" : "+v"(old) :: "memory");
            unsigned seen = __builtin_amdgcn_readfirstlane(old);
            while (seen & (1u << want)) {
                want = __builtin_ctz(~seen & 0xffu);
                unsigned o2 = 0;
                if (lane == 0) o2 = __hip_atomic_fetch_or(mask, 1u << want, __ATOMIC_RELAXED, __HIP_MEMORY_SCOPE_AGENT);
                seen = __builtin_amdgcn_readfirstlane(o2);
            }
            if (lane == 0) {
                *(LAS unsigned*)(l + LDS_CTL + 4) = c; *(LAS unsigned*)(l + LDS_CTL + 8) = 0; *(LAS unsigned*)(l + LDS_CTL + 12) = want;
            }
            asm volatile("s_waitcnt lgkmcnt(0)" ::: "memory");
        } else {
            issueB(0, 0); issueB(1, 1);
            waitv<4>();
        }
    } else {
        issueB(0, 0); issueB(1, 1);
    }
    bar();
    const unsigned call = __builtin_amdgcn_readfirstlane(*(LAS unsigned*)(l + LDS_CTL + 4));
    const int q = __builtin_amdgcn_readfirstlane(*(LAS unsigned*)(l + LDS_CTL + 12));
    const int src = (p.rank + q) & 7;
    if (wid == 1 && lane == 0) {
        const unsigned fo = (unsigned)(OFF_FLAG + p.rank * 1024 + b * 4);
        char* const pw[7] = {w1, w2, w3, w4, w5, w6, w7};
#pragma unroll
        for (int i = 0; i < 7; ++i)
            __hip_atomic_store((GAS unsigned*)(pw[i] + fo), call, __ATOMIC_RELAXED, __HIP_MEMORY_SCOPE_SYSTEM);
        __hip_atomic_store((GAS unsigned*)(self + OFF_CNT + b * 128), call, __ATOMIC_RELAXED, __HIP_MEMORY_SCOPE_AGENT);
        if ((b & 7) == 0)
            __hip_atomic_store((GAS unsigned*)(self + OFF_CLAIM + (((call + 1) & 1) * 32 + ti) * 128), 0u, __ATOMIC_RELAXED, __HIP_MEMORY_SCOPE_AGENT);
    }

    const unsigned char* Abase; unsigned RS, KS;
    const bool remote = q != 0;
    if (!remote) { Abase = p.x; RS = KD; KS = BK; }
    else { Abase = (const unsigned char*)(p.ws[q] + OFF_STG + (call & 1) * STG_BYTES); RS = BK; KS = MSH * BK; }
    Abase = uni(Abase);
    unsigned voffA[4];
#pragma unroll
    for (int j = 0; j < 4; ++j) {
        const int pa = wid * 4 + j, row = pa * 8 + (lane >> 3);
        const int ch = (lane & 7) ^ ((pa & 1) * 4 + (lane >> 4));
        voffA[j] = (unsigned)(rtile * BM + row) * RS + ch * 16;
    }
    auto issueA = [&](int kt, int slot) {
        const unsigned char* base = Abase + (size_t)kt * KS;
#pragma unroll
        for (int j = 0; j < 4; ++j) dma16(lbase + slot * ST + (wid * 4 + j) * 1024, voffA[j], base);
    };
    GAS unsigned* tmo = (GAS unsigned*)(self + OFF_TMO);
    const unsigned char* fbase = uni(self + OFF_FLAG + src * 1024);
    auto gave_up = [&](unsigned long long t0) -> bool {
        if (__builtin_amdgcn_readfirstlane(__hip_atomic_load(tmo, __ATOMIC_RELAXED, __HIP_MEMORY_SCOPE_AGENT)) != call && rt() - t0 <= DEADLINE_TICKS) return false;
        if (lane == 0) __hip_atomic_store(tmo, call, __ATOMIC_RELAXED, __HIP_MEMORY_SCOPE_AGENT);
        return true;
    };
    bool ok = true;
    if (remote) {
        if (wid == 0) {
            const unsigned pv = (lane & 15) * 4;
            const unsigned long long t0 = rt();
            unsigned good = 1;
#pragma unroll
            for (int i = 0; i < 3; ++i) { dma4_sys(lbase + LDS_POLL + i * 256, pv, fbase); __builtin_amdgcn_s_sleep(5); }
            for (unsigned it = 0;; ++it) {
                dma4_sys(lbase + LDS_POLL + ((it + 3) & 3) * 256, pv, fbase);
                waitv<3>();
                const unsigned f = *(volatile LAS unsigned*)(l + LDS_POLL + (it & 3) * 256 + lane * 4);
                if (__all(f >= call)) break;
                if ((it & 63) == 63 && gave_up(t0)) { good = 0; break; }
            }
            if (lane == 0) *(LAS unsigned*)(l + LDS_CTL) = good;
            asm volatile("s_waitcnt lgkmcnt(0)" ::: "memory");
        }
        bar();
        ok = __builtin_amdgcn_readfirstlane(*(LAS unsigned*)(l + LDS_CTL)) != 0;
    }
    if (ok) {
        issueA(0, 0);
        if (remote && wid == 0) dma16_sys(lbase + LDS_FLAGS, lane * 16, fbase);
        issueA(1, 1);
    }

    const int wm = wid >> 1, wn = wid & 1;
    const unsigned sA = (lane & 15) >> 1, g4 = lane >> 4;
    const unsigned aA0 = (unsigned)(wm * 64 + (lane & 15)) * 128 + ((g4 ^ sA) * 16), aA1 = aA0 ^ 64;
    const unsigned rB = (lane & 15) >> 1, sB = ((rB >> 1) & 3) | ((g4 & 1) << 2);
    unsigned aB[4];
#pragma unroll
    for (int ni = 0; ni < 4; ++ni) aB[ni] = A_ST + (16 * g4 + rB) * 128 + (((wn * 4 + ni) ^ sB) * 16) + (lane & 1) * 8;

    v4f acc[4][4];
#pragma unroll
    for (int mi = 0; mi < 4; ++mi)
#pragma unroll
        for (int ni = 0; ni < 4; ++ni) acc[mi][ni] = (v4f){0.f, 0.f, 0.f, 0.f};

    if (ok) {
        waitv<4>();
        bar();
        if (remote) {
            const v4u f = *(LAS v4u*)(l + LDS_FLAGS + lane * 16);
            if (!__all(f.x >= call && f.y >= call && f.z >= call && f.w >= call)) {
                if (wid == 0) {
                    const char* fp = (const char*)fbase + lane * 16;
                    const unsigned long long t0 = rt();
                    unsigned good = 1;
                    for (unsigned it = 1;; ++it) {
                        v4u h;
                        asm volatile("global_load_dwordx4 %0, %1, off sc0 sc1\n\ts_waitcnt vmcnt(0)" : "=&v"(h) : "v"(fp) : "memory");
                        if (__all(h.x >= call && h.y >= call && h.z >= call && h.w >= call)) break;
                        if ((it & 15) == 0 && gave_up(t0)) { good = 0; break; }
                        __builtin_amdgcn_s_sleep(2);
                    }
                    if (lane == 0) *(LAS unsigned*)(l + LDS_CTL) = good;
                    asm volatile("s_waitcnt lgkmcnt(0)" ::: "memory");
                }
                bar();
                ok = __builtin_amdgcn_readfirstlane(*(LAS unsigned*)(l + LDS_CTL)) != 0;
            }
        }
    }

    const int orow = src * MSH + rtile * BM + wm * 64 + (lane & 7), ocol = ct * BN + wn * 64 + (lane & 8) * 2 + 4 * (lane >> 4);
    float* const optr = p.out + (size_t)orow * NL + ocol;
    auto store_rows = [&](int mi, float sc) {
#pragma unroll
        for (int np = 0; np < 2; ++np) {
            const v4f a = acc[mi][2 * np] * sc, bb = acc[mi][2 * np + 1] * sc;
            v4f x, y;
#pragma unroll
            for (int j = 0; j < 4; ++j) {
                const int aj = __float_as_int(a[j]), bj = __float_as_int(bb[j]);
                x[j] = __int_as_float(__builtin_amdgcn_update_dpp(aj, bj, 0x128, 0xf, 0xc, false));
                y[j] = __int_as_float(__builtin_amdgcn_update_dpp(bj, aj, 0x128, 0xf, 0x3, false));
            }
            float* q = optr + (size_t)mi * 16 * NL + np * 32;
            asm volatile("global_store_dwordx4 %0, %1, off sc1\n\ts_nop 1" :: "v"(q), "v"(x) : "memory");
            asm volatile("global_store_dwordx4 %0, %1, off sc1\n\ts_nop 1" :: "v"(q + 8 * NL), "v"(y) : "memory");
        }
    };

    if (ok) {
        int slot = 0;
        v8i Af[4], Bf[4];
        auto read_frags = [&]() {
            const unsigned so = slot * ST;
#pragma unroll
            for (int ni = 0; ni < 4; ++ni) {
                LAS char* q = l + so + aB[ni];
                const v2i b0 = __builtin_amdgcn_ds_read_tr8_b64_v2i32((LAS v2i*)(q));
                const v2i b1 = __builtin_amdgcn_ds_read_tr8_b64_v2i32((LAS v2i*)(q + 1024));
                const v2i b2 = __builtin_amdgcn_ds_read_tr8_b64_v2i32((LAS v2i*)(q + 8192));
                const v2i b3 = __builtin_amdgcn_ds_read_tr8_b64_v2i32((LAS v2i*)(q + 9216));
                Bf[ni] = (v8i){b0.x, b0.y, b1.x, b1.y, b2.x, b2.y, b3.x, b3.y};
            }
#pragma unroll
            for (int mi = 0; mi < 4; ++mi) {
                const v4i a0 = *(LAS v4i*)(l + so + aA0 + mi * 2048);
                const v4i a1 = *(LAS v4i*)(l + so + aA1 + mi * 2048);
                Af[mi] = (v8i){a0.x, a0.y, a0.z, a0.w, a1.x, a1.y, a1.z, a1.w};
            }
        };
        auto mma_row = [&](int mi) {
#pragma unroll
            for (int ni = 0; ni < 4; ++ni)
                acc[mi][ni] = __builtin_amdgcn_mfma_scale_f32_16x16x128_f8f6f4(Bf[ni], Af[mi], acc[mi][ni], 0, 0, 0, 0, 0, 0);
        };
        auto next_slot = [&]() { slot = slot == 2 ? 0 : slot + 1; };
        auto pieceA = [&](int j, int kt, int sl) { dma16(lbase + sl * ST + (wid * 4 + j) * 1024, voffA[j], Abase + (size_t)kt * KS); };
        auto pieceB = [&](int j, int kt, int sl) { dma16(lbase + sl * ST + A_ST + (wid * 2 + j) * 1024, voffB[j], p.w + (size_t)kt * BK * NL); };
        auto tile = [&](auto&& issue) {
            read_frags();
#pragma unroll
            for (int mi = 0; mi < 4; ++mi) {
#pragma unroll
                for (int ni = 0; ni < 4; ++ni) {
                    acc[mi][ni] = __builtin_amdgcn_mfma_scale_f32_16x16x128_f8f6f4(Bf[ni], Af[mi], acc[mi][ni], 0, 0, 0, 0, 0, 0);
                    const int i = mi * 4 + ni;
                    if ((i & 1) == 1 && i < 12) {
                        __builtin_amdgcn_sched_barrier(0);
                        issue(i >> 1);
                        __builtin_amdgcn_sched_barrier(0);
                    }
                }
            }
            asm volatile("s_waitcnt lgkmcnt(0)" ::: "memory");
        };
        auto last_tile = [&](float sc) {
            read_frags();
            mma_row(0); mma_row(1);
            __builtin_amdgcn_sched_barrier(0);
            store_rows(0, sc);
            __builtin_amdgcn_sched_barrier(0);
            mma_row(2);
            __builtin_amdgcn_sched_barrier(0);
            store_rows(1, sc);
            __builtin_amdgcn_sched_barrier(0);
            mma_row(3);
            __builtin_amdgcn_sched_barrier(0);
            store_rows(2, sc);
            store_rows(3, sc);
        };
        for (int k = 0; k < NT - 5; ++k) {
            const int s2 = slot == 0 ? 2 : slot - 1;
            tile([&](int j) { if (j < 4) pieceA(j, k + 2, s2); else pieceB(j - 4, k + 2, s2); });
            waitv<6>();
            bar();
            next_slot();
        }
        static_assert(NT == 32, "");
        unsigned tmov = __hip_atomic_load(tmo, __ATOMIC_RELAXED, __HIP_MEMORY_SCOPE_AGENT);
        LAS unsigned* hb = (LAS unsigned*)(l + LDS_CTL + 8);
        if (wid < 4) {
            tile([&](int j) { if (j < 4) pieceA(j, 29, 2); else pieceB(j - 4, 29, 2); });
            waitv<6>(); bar(); next_slot();
            tile([&](int j) { if (j < 4) pieceA(j, 30, 0); else pieceB(j - 4, 30, 0); });
            waitv<6>(); bar(); next_slot();
            tile([&](int j) { if (j < 4) pieceA(j, 31, 1); else pieceB(j - 4, 31, 1); });
            waitv<6>(); bar(); next_slot();
            tile([&](int) {});
            waitv<0>(); bar(); next_slot();
            asm volatile("" : "+v"(tmov));
            const float sc = __builtin_amdgcn_readfirstlane(tmov) != call ? scale : __builtin_nanf("");
            last_tile(sc);
        } else {
            tile([&](int j) { if (j >= 4) pieceB(j - 4, 29, 2); });
            waitv<2>(); bar(); next_slot();
            tile([&](int j) { if (j < 4) pieceA(j, 29, 2); else pieceB(j - 4, 30, 0); });
            waitv<6>(); bar();
            pieceB(0, 31, 1); pieceB(1, 31, 1);
            __builtin_amdgcn_s_sleep(6);
#pragma unroll
            for (int j = 0; j < 4; ++j) pieceA(j, 30, 0);
            waitv<6>(); bar();
            slot = 2;
            tile([&](int j) { if (j < 4) pieceA(j, 31, 1); });
            waitv<8>(); bar();
            auto half_bar = [&](unsigned target) {
                if (lane == 0) __hip_atomic_fetch_add(hb, 1u, __ATOMIC_RELAXED, __HIP_MEMORY_SCOPE_WORKGROUP);
                while (__builtin_amdgcn_readfirstlane(__hip_atomic_load(hb, __ATOMIC_RELAXED, __HIP_MEMORY_SCOPE_WORKGROUP)) < target) __builtin_amdgcn_s_sleep(0);
                asm volatile("" ::: "memory");
            };
            waitv<4>(); half_bar(4);
            slot = 0;
            tile([&](int) {});
            waitv<0>(); half_bar(8);
            slot = 1;
            asm volatile("" : "+v"(tmov));
            const float sc = __builtin_amdgcn_readfirstlane(tmov) != call ? scale : __builtin_nanf("");
            last_tile(sc);
        }
    } else {
        waitv<0>();
#pragma unroll
        for (int mi = 0; mi < 4; ++mi) store_rows(mi, __builtin_nanf(""));
    }
}
}

extern "C" void kernel_launch(int rank, int nranks, ncclComm_t comm, hipStream_t stream,
                              void* const* d_in, const int* in_sizes, int n_in,
                              void* d_out, int out_size, const int* in_dtypes, int out_dtype,
                              void* const* d_out_all, void* const* d_ws_all, size_t ws_bytes) {
    using namespace agg;
    (void)comm; (void)in_dtypes; (void)out_dtype; (void)d_out_all;
    if (nranks != NR || n_in != 4 || (unsigned)rank >= (unsigned)NR || ws_bytes < WS_NEED ||
        in_sizes[0] != MSH * KD || in_sizes[1] != KD * NL || out_size != NR * MSH * NL) {
        fprintf(stderr, "ag_gemm: unexpected problem shape\n");
        return;
    }
    hipError_t e = hipFuncSetAttribute((const void*)ag_gemm, hipFuncAttributeMaxDynamicSharedMemorySize, LDS_BYTES);
    if (e != hipSuccess) { fprintf(stderr, "ag_gemm: hipFuncSetAttribute failed: %s\n", hipGetErrorString(e)); return; }
    Params p;
    p.x = (const unsigned char*)d_in[0]; p.w = (const unsigned char*)d_in[1];
    p.sx = (const float*)d_in[2]; p.sw = (const float*)d_in[3]; p.out = (float*)d_out;
    for (int i = 0; i < NR; ++i) p.ws[i] = (char*)d_ws_all[(rank + i) & 7];
    p.rank = rank;
    static const unsigned PERM[NR] = {0x51473602u, 0x12570364u, 0x72536104u, 0x26413705u, 0x74261053u, 0x53164702u, 0x76135240u, 0x14370625u};
    p.perm = PERM[rank];
    {
        static int once = 0;
        if (!once) {
            once = 1;
            int dev = -1, n = -1; char bus[64] = {0};
            (void)hipGetDevice(&dev); (void)hipGetDeviceCount(&n); (void)hipDeviceGetPCIBusId(bus, 64, dev);
            char line[512]; int o = snprintf(line, sizeof line, "AGGDBG rank %d dev %d of %d pci %s owners:", rank, dev, n, bus);
            for (int i = 0; i < NR; ++i) {
                hipPointerAttribute_t a; hipError_t r = hipPointerGetAttributes(&a, d_ws_all[i]);
                char b2[64] = "?"; if (r == hipSuccess && a.device >= 0 && a.device < n) (void)hipDeviceGetPCIBusId(b2, 64, a.device);
                o += snprintf(line + o, sizeof line - o, " [%d: err %d dev %d %s]", i, (int)r, r == hipSuccess ? a.device : -1, b2);
            }
            (void)hipGetLastError();
            fprintf(stderr, "%s\n", line);
        }
    }
    hipLaunchKernelGGL(ag_gemm, dim3(256), dim3(512), LDS_BYTES, stream, p);
    e = hipGetLastError();
    if (e != hipSuccess) fprintf(stderr, "ag_gemm: launch failed: %s\n", hipGetErrorString(e));
}
```

```cpp
#include <hip/hip_runtime.h>
#include <rccl/rccl.h>
#include <cstdio>

namespace agg {
#define LAS __attribute__((address_space(3)))
#define GAS __attribute__((address_space(1)))
typedef int v8i __attribute__((ext_vector_type(8)));
typedef int v4i __attribute__((ext_vector_type(4)));
typedef int v2i __attribute__((ext_vector_type(2)));
typedef unsigned v4u __attribute__((ext_vector_type(4)));
typedef float v4f __attribute__((ext_vector_type(4)));

constexpr int NR = 8;
constexpr int MSH = 512, KD = 4096, NL = 2048;
constexpr int BM = 256, BN = 128, BK = 128, NT = KD / BK;
constexpr int A_ST = BM * BK, B_ST = BK * BN, ST = A_ST + B_ST, NSTAGE = 3;
constexpr int LDS_POLL = NSTAGE * ST;
constexpr int LDS_FLAGS = LDS_POLL + 1024;
constexpr int LDS_CTL = LDS_FLAGS + 1024;
constexpr int LDS_BYTES = LDS_CTL + 64;
constexpr size_t OFF_CNT = 0;
constexpr size_t OFF_TMO = 32768;
constexpr size_t OFF_CLAIM = 49152;
constexpr size_t OFF_FLAG = 65536;
constexpr size_t OFF_STG = 1u << 20;
constexpr size_t STG_BYTES = (size_t)MSH * KD;
constexpr size_t WS_NEED = OFF_STG + 2 * STG_BYTES;
constexpr unsigned long long DEADLINE_TICKS = 50000000ull;

struct Params {
    const unsigned char* x; const unsigned char* w; const float* sx; const float* sw; float* out;
    char* ws[NR];
    int rank; unsigned perm;
};

__device__ __forceinline__ unsigned long long rt() { return __builtin_amdgcn_s_memrealtime(); }
template <int N> __device__ __forceinline__ void waitv() { asm volatile("s_waitcnt vmcnt(%0)" :: "n"(N) : "memory"); }
__device__ __forceinline__ void dma16(unsigned ldsaddr, unsigned voff, const void* base) {
    asm volatile("s_mov_b32 m0, %0\n\ts_nop 0\n\tglobal_load_lds_dwordx4 %1, %2" :: "s"(ldsaddr), "v"(voff), "s"(base) : "memory");
}
__device__ __forceinline__ void dma4_sys(unsigned ldsaddr, unsigned voff, const void* base) {
    asm volatile("s_mov_b32 m0, %0\n\ts_nop 0\n\tglobal_load_lds_dword %1, %2 sc0 sc1" :: "s"(ldsaddr), "v"(voff), "s"(base) : "memory");
}
__device__ __forceinline__ void dma16_sys(unsigned ldsaddr, unsigned voff, const void* base) {
    asm volatile("s_mov_b32 m0, %0\n\ts_nop 0\n\tglobal_load_lds_dwordx4 %1, %2 sc0 sc1" :: "s"(ldsaddr), "v"(voff), "s"(base) : "memory");
}
__device__ __forceinline__ const unsigned char* uni(const void* p) {
    const unsigned long long v = (unsigned long long)p;
    const unsigned lo = __builtin_amdgcn_readfirstlane((unsigned)v), hi = __builtin_amdgcn_readfirstlane((unsigned)(v >> 32));
    return (const unsigned char*)(((unsigned long long)hi << 32) | lo);
}
__device__ __forceinline__ void bar() { asm volatile("" ::: "memory"); __builtin_amdgcn_s_barrier(); asm volatile("" ::: "memory"); }


__global__ __launch_bounds__(512, 2) void ag_gemm(Params p) {
    extern __shared__ __attribute__((aligned(16))) char smem[];
    LAS char* l = (LAS char*)smem;
    const unsigned lbase = (unsigned)(size_t)l;
    const int tid = threadIdx.x, lane = tid & 63;
    const int wid = __builtin_amdgcn_readfirstlane(tid >> 6);
    const int b = blockIdx.x, ti = b >> 3, rtile = ti & 1, ct = ti >> 1;
    char* const self = p.ws[0];
    char* const w1 = p.ws[1]; char* const w2 = p.ws[2]; char* const w3 = p.ws[3]; char* const w4 = p.ws[4];
    char* const w5 = p.ws[5]; char* const w6 = p.ws[6]; char* const w7 = p.ws[7];
    asm volatile("" :: "s"(self), "s"(w1), "s"(w2), "s"(w3), "s"(w4), "s"(w5), "s"(w6), "s"(w7), "s"(p.x), "s"(p.w));
    typedef const __attribute__((address_space(4))) float* cfp;
    const float scale = *(cfp)p.sx * *(cfp)p.sw;

    unsigned voffB[2];
#pragma unroll
    for (int j = 0; j < 2; ++j) {
        const int pb = wid * 2 + j, k = pb * 8 + (lane >> 3);
        const int ch = (lane & 7) ^ (((lane >> 4) & 3) | (((pb >> 1) & 1) << 2));
        voffB[j] = (unsigned)(k * NL + ct * BN + ch * 16);
    }
    auto issueB = [&](int kt, int slot) {
        const unsigned char* base = p.w + (size_t)kt * BK * NL;
#pragma unroll
        for (int j = 0; j < 2; ++j) dma16(lbase + slot * ST + A_ST + (wid * 2 + j) * 1024, voffB[j], base);
    };

    if (wid < 4) {
        const int skt = b >> 3, sch = tid & 7, srow = (b & 7) * 64 + (tid >> 3);
        const unsigned char* sp = p.x + (size_t)srow * KD + skt * BK + sch * 16;
        const v4u sv0 = *(const GAS v4u*)sp, sv1 = *(const GAS v4u*)(sp + 32 * KD);
        const unsigned cnt = __hip_atomic_load((GAS unsigned*)(self + OFF_CNT + b * 128), __ATOMIC_RELAXED, __HIP_MEMORY_SCOPE_AGENT);
        const unsigned c = __builtin_amdgcn_readfirstlane(cnt) + 1;
        char* dst = self + OFF_STG + (c & 1) * STG_BYTES + (size_t)skt * (MSH * BK) + srow * BK + sch * 16;
        asm volatile("global_store_dwordx4 %0, %1, off sc0 sc1\n\tglobal_store_dwordx4 %2, %3, off sc0 sc1\n\ts_nop 1" :: "v"(dst), "v"(sv0), "v"(dst + 32 * BK), "v"(sv1) : "memory");
        if (wid == 0) {
            const unsigned xcc = (unsigned)__builtin_amdgcn_s_getreg(20 | (2 << 11)) & 7u;
            unsigned want = (p.perm >> (4 * xcc)) & 7u;
            GAS unsigned* mask = (GAS unsigned*)(self + OFF_CLAIM + ((c & 1) * 32 + ti) * 128);
            unsigned old = 0;
            if (lane == 0) asm volatile("global_atomic_or %0, %1, %2, off sc0" : "+v"(old) : "v"(mask), "v"(1u << want) : "memory");
            issueB(0, 0); issueB(1, 1);
            asm volatile("s_waitcnt vmcnt(4)" : "+v"(old) :: "memory");
            unsigned seen = __builtin_amdgcn_readfirstlane(old);
            while (seen & (1u << want)) {
                want = __builtin_ctz(~seen & 0xffu);
                unsigned o2 = 0;
                if (lane == 0) o2 = __hip_atomic_fetch_or(mask, 1u << want, __ATOMIC_RELAXED, __HIP_MEMORY_SCOPE_AGENT);
                seen = __builtin_amdgcn_readfirstlane(o2);
            }
            if (lane == 0) {
                *(LAS unsigned*)(l + LDS_CTL + 4) = c; *(LAS unsigned*)(l + LDS_CTL + 8) = 0; *(LAS unsigned*)(l + LDS_CTL + 12) = want;
            }
            asm volatile("s_waitcnt lgkmcnt(0)" ::: "memory");
        } else {
            issueB(0, 0); issueB(1, 1);
            waitv<4>();
        }
    } else {
        issueB(0, 0); issueB(1, 1);
    }
    bar();
    const unsigned call = __builtin_amdgcn_readfirstlane(*(LAS unsigned*)(l + LDS_CTL + 4));
    const int q = __builtin_amdgcn_readfirstlane(*(LAS unsigned*)(l + LDS_CTL + 12));
    const int src = (p.rank + q) & 7;
    if (wid == 1 && lane == 0) {
        const unsigned fo = (unsigned)(OFF_FLAG + p.rank * 1024 + b * 4);
        char* const pw[7] = {w1, w2, w3, w4, w5, w6, w7};
#pragma unroll
        for (int i = 0; i < 7; ++i)
            __hip_atomic_store((GAS unsigned*)(pw[i] + fo), call, __ATOMIC_RELAXED, __HIP_MEMORY_SCOPE_SYSTEM);
        __hip_atomic_store((GAS unsigned*)(self + OFF_CNT + b * 128), call, __ATOMIC_RELAXED, __HIP_MEMORY_SCOPE_AGENT);
        if ((b & 7) == 0)
            __hip_atomic_store((GAS unsigned*)(self + OFF_CLAIM + (((call + 1) & 1) * 32 + ti) * 128), 0u, __ATOMIC_RELAXED, __HIP_MEMORY_SCOPE_AGENT);
    }

    const unsigned char* Abase; unsigned RS, KS;
    const bool remote = q != 0;
    if (!remote) { Abase = p.x; RS = KD; KS = BK; }
    else { Abase = (const unsigned char*)(p.ws[q] + OFF_STG + (call & 1) * STG_BYTES); RS = BK; KS = MSH * BK; }
    Abase = uni(Abase);
    unsigned voffA[4];
#pragma unroll
    for (int j = 0; j < 4; ++j) {
        const int pa = wid * 4 + j, row = pa * 8 + (lane >> 3);
        const int ch = (lane & 7) ^ ((pa & 1) * 4 + (lane >> 4));
        voffA[j] = (unsigned)(rtile * BM + row) * RS + ch * 16;
    }
    auto issueA = [&](int kt, int slot) {
        const unsigned char* base = Abase + (size_t)kt * KS;
#pragma unroll
        for (int j = 0; j < 4; ++j) dma16(lbase + slot * ST + (wid * 4 + j) * 1024, voffA[j], base);
    };
    GAS unsigned* tmo = (GAS unsigned*)(self + OFF_TMO);
    const unsigned char* fbase = uni(self + OFF_FLAG + src * 1024);
    auto gave_up = [&](unsigned long long t0) -> bool {
        if (__builtin_amdgcn_readfirstlane(__hip_atomic_load(tmo, __ATOMIC_RELAXED, __HIP_MEMORY_SCOPE_AGENT)) != call && rt() - t0 <= DEADLINE_TICKS) return false;
        if (lane == 0) __hip_atomic_store(tmo, call, __ATOMIC_RELAXED, __HIP_MEMORY_SCOPE_AGENT);
        return true;
    };
    bool ok = true;
    if (remote) {
        if (wid == 0) {
            const unsigned pv = (lane & 15) * 4;
            const unsigned long long t0 = rt();
            unsigned good = 1;
#pragma unroll
            for (int i = 0; i < 3; ++i) { dma4_sys(lbase + LDS_POLL + i * 256, pv, fbase); __builtin_amdgcn_s_sleep(5); }
            for (unsigned it = 0;; ++it) {
                dma4_sys(lbase + LDS_POLL + ((it + 3) & 3) * 256, pv, fbase);
                waitv<3>();
                const unsigned f = *(volatile LAS unsigned*)(l + LDS_POLL + (it & 3) * 256 + lane * 4);
                if (__all(f >= call)) break;
                if ((it & 63) == 63 && gave_up(t0)) { good = 0; break; }
            }
            if (lane == 0) *(LAS unsigned*)(l + LDS_CTL) = good;
            asm volatile("s_waitcnt lgkmcnt(0)" ::: "memory");
        }
        bar();
        ok = __builtin_amdgcn_readfirstlane(*(LAS unsigned*)(l + LDS_CTL)) != 0;
    }
    if (ok) {
        issueA(0, 0);
        if (remote && wid == 0) dma16_sys(lbase + LDS_FLAGS, lane * 16, fbase);
        issueA(1, 1);
    }

    const int wm = wid >> 1, wn = wid & 1;
    const unsigned sA = (lane & 15) >> 1, g4 = lane >> 4;
    const unsigned aA0 = (unsigned)(wm * 64 + (lane & 15)) * 128 + ((g4 ^ sA) * 16), aA1 = aA0 ^ 64;
    const unsigned rB = (lane & 15) >> 1, sB = ((rB >> 1) & 3) | ((g4 & 1) << 2);
    unsigned aB[4];
#pragma unroll
    for (int ni = 0; ni < 4; ++ni) aB[ni] = A_ST + (16 * g4 + rB) * 128 + (((wn * 4 + ni) ^ sB) * 16) + (lane & 1) * 8;

    v4f acc[4][4];
#pragma unroll
    for (int mi = 0; mi < 4; ++mi)
#pragma unroll
        for (int ni = 0; ni < 4; ++ni) acc[mi][ni] = (v4f){0.f, 0.f, 0.f, 0.f};

    if (ok) {
        waitv<4>();
        bar();
        if (remote) {
            const v4u f = *(LAS v4u*)(l + LDS_FLAGS + lane * 16);
            if (!__all(f.x >= call && f.y >= call && f.z >= call && f.w >= call)) {
                if (wid == 0) {
                    const char* fp = (const char*)fbase + lane * 16;
                    const unsigned long long t0 = rt();
                    unsigned good = 1;
                    for (unsigned it = 1;; ++it) {
                        v4u h;
                        asm volatile("global_load_dwordx4 %0, %1, off sc0 sc1\n\ts_waitcnt vmcnt(0)" : "=&v"(h) : "v"(fp) : "memory");
                        if (__all(h.x >= call && h.y >= call && h.z >= call && h.w >= call)) break;
                        if ((it & 15) == 0 && gave_up(t0)) { good = 0; break; }
                        __builtin_amdgcn_s_sleep(2);
                    }
                    if (lane == 0) *(LAS unsigned*)(l + LDS_CTL) = good;
                    asm volatile("s_waitcnt lgkmcnt(0)" ::: "memory");
                }
                bar();
                ok = __builtin_amdgcn_readfirstlane(*(LAS unsigned*)(l + LDS_CTL)) != 0;
            }
        }
    }

    const int orow = src * MSH + rtile * BM + wm * 64 + (lane & 7), ocol = ct * BN + wn * 64 + (lane & 8) * 2 + 4 * (lane >> 4);
    float* const optr = p.out + (size_t)orow * NL + ocol;
    auto store_rows = [&](int mi, float sc) {
#pragma unroll
        for (int np = 0; np < 2; ++np) {
            const v4f a = acc[mi][2 * np] * sc, bb = acc[mi][2 * np + 1] * sc;
            v4f x, y;
#pragma unroll
            for (int j = 0; j < 4; ++j) {
                const int aj = __float_as_int(a[j]), bj = __float_as_int(bb[j]);
                x[j] = __int_as_float(__builtin_amdgcn_update_dpp(aj, bj, 0x128, 0xf, 0xc, false));
                y[j] = __int_as_float(__builtin_amdgcn_update_dpp(bj, aj, 0x128, 0xf, 0x3, false));
            }
            float* q = optr + (size_t)mi * 16 * NL + np * 32;
            asm volatile("global_store_dwordx4 %0, %1, off sc1\n\ts_nop 1" :: "v"(q), "v"(x) : "memory");
            asm volatile("global_store_dwordx4 %0, %1, off sc1\n\ts_nop 1" :: "v"(q + 8 * NL), "v"(y) : "memory");
        }
    };

    if (ok) {
        int slot = 0;
        v8i Af[4], Bf[4];
        auto read_frags = [&]() {
            const unsigned so = slot * ST;
#pragma unroll
            for (int ni = 0; ni < 4; ++ni) {
                LAS char* q = l + so + aB[ni];
                const v2i b0 = __builtin_amdgcn_ds_read_tr8_b64_v2i32((LAS v2i*)(q));
                const v2i b1 = __builtin_amdgcn_ds_read_tr8_b64_v2i32((LAS v2i*)(q + 1024));
                const v2i b2 = __builtin_amdgcn_ds_read_tr8_b64_v2i32((LAS v2i*)(q + 8192));
                const v2i b3 = __builtin_amdgcn_ds_read_tr8_b64_v2i32((LAS v2i*)(q + 9216));
                Bf[ni] = (v8i){b0.x, b0.y, b1.x, b1.y, b2.x, b2.y, b3.x, b3.y};
            }
#pragma unroll
            for (int mi = 0; mi < 4; ++mi) {
                const v4i a0 = *(LAS v4i*)(l + so + aA0 + mi * 2048);
                const v4i a1 = *(LAS v4i*)(l + so + aA1 + mi * 2048);
                Af[mi] = (v8i){a0.x, a0.y, a0.z, a0.w, a1.x, a1.y, a1.z, a1.w};
            }
        };
        auto mma_row = [&](int mi) {
#pragma unroll
            for (int ni = 0; ni < 4; ++ni)
                acc[mi][ni] = __builtin_amdgcn_mfma_scale_f32_16x16x128_f8f6f4(Bf[ni], Af[mi], acc[mi][ni], 0, 0, 0, 0, 0, 0);
        };
        auto next_slot = [&]() { slot = slot == 2 ? 0 : slot + 1; };
        auto pieceA = [&](int j, int kt, int sl) { dma16(lbase + sl * ST + (wid * 4 + j) * 1024, voffA[j], Abase + (size_t)kt * KS); };
        auto pieceB = [&](int j, int kt, int sl) { dma16(lbase + sl * ST + A_ST + (wid * 2 + j) * 1024, voffB[j], p.w + (size_t)kt * BK * NL); };
        auto tile = [&](auto&& issue) {
            read_frags();
#pragma unroll
            for (int mi = 0; mi < 4; ++mi) {
#pragma unroll
                for (int ni = 0; ni < 4; ++ni) {
                    acc[mi][ni] = __builtin_amdgcn_mfma_scale_f32_16x16x128_f8f6f4(Bf[ni], Af[mi], acc[mi][ni], 0, 0, 0, 0, 0, 0);
                    const int i = mi * 4 + ni;
                    if ((i & 1) == 1 && i < 12) {
                        __builtin_amdgcn_sched_barrier(0);
                        issue(i >> 1);
                        __builtin_amdgcn_sched_barrier(0);
                    }
                }
            }
            asm volatile("s_waitcnt lgkmcnt(0)" ::: "memory");
        };
        auto last_tile = [&](float sc) {
            read_frags();
            mma_row(0); mma_row(1);
            __builtin_amdgcn_sched_barrier(0);
            store_rows(0, sc);
            __builtin_amdgcn_sched_barrier(0);
            mma_row(2);
            __builtin_amdgcn_sched_barrier(0);
            store_rows(1, sc);
            __builtin_amdgcn_sched_barrier(0);
            mma_row(3);
            __builtin_amdgcn_sched_barrier(0);
            store_rows(2, sc);
            store_rows(3, sc);
        };
        for (int k = 0; k < NT - 5; ++k) {
            const int s2 = slot == 0 ? 2 : slot - 1;
            tile([&](int j) { if (j < 4) pieceA(j, k + 2, s2); else pieceB(j - 4, k + 2, s2); });
            waitv<6>();
            bar();
            next_slot();
        }
        static_assert(NT == 32, "");
        unsigned tmov = __hip_atomic_load(tmo, __ATOMIC_RELAXED, __HIP_MEMORY_SCOPE_AGENT);
        LAS unsigned* hb = (LAS unsigned*)(l + LDS_CTL + 8);
        if (wid < 4) {
            tile([&](int j) { if (j < 4) pieceA(j, 29, 2); else pieceB(j - 4, 29, 2); });
            waitv<6>(); bar(); next_slot();
            tile([&](int j) { if (j < 4) pieceA(j, 30, 0); else pieceB(j - 4, 30, 0); });
            waitv<6>(); bar(); next_slot();
            tile([&](int j) { if (j < 4) pieceA(j, 31, 1); else pieceB(j - 4, 31, 1); });
            waitv<6>(); bar(); next_slot();
            tile([&](int) {});
            waitv<0>(); bar(); next_slot();
            asm volatile("" : "+v"(tmov));
            const float sc = __builtin_amdgcn_readfirstlane(tmov) != call ? scale : __builtin_nanf("");
            last_tile(sc);
        } else {
            tile([&](int j) { if (j >= 4) pieceB(j - 4, 29, 2); });
            waitv<2>(); bar(); next_slot();
            tile([&](int j) { if (j < 4) pieceA(j, 29, 2); else pieceB(j - 4, 30, 0); });
            waitv<6>(); bar();
            pieceB(0, 31, 1); pieceB(1, 31, 1);
            __builtin_amdgcn_s_sleep(6);
#pragma unroll
            for (int j = 0; j < 4; ++j) pieceA(j, 30, 0);
            waitv<6>(); bar();
            slot = 2;
            tile([&](int j) { if (j < 4) pieceA(j, 31, 1); });
            waitv<8>(); bar();
            auto half_bar = [&](unsigned target) {
                if (lane == 0) __hip_atomic_fetch_add(hb, 1u, __ATOMIC_RELAXED, __HIP_MEMORY_SCOPE_WORKGROUP);
                while (__builtin_amdgcn_readfirstlane(__hip_atomic_load(hb, __ATOMIC_RELAXED, __HIP_MEMORY_SCOPE_WORKGROUP)) < target) __builtin_amdgcn_s_sleep(0);
                asm volatile("" ::: "memory");
            };
            waitv<4>(); half_bar(4);
            slot = 0;
            tile([&](int) {});
            waitv<0>(); half_bar(8);
            slot = 1;
            asm volatile("" : "+v"(tmov));
            const float sc = __builtin_amdgcn_readfirstlane(tmov) != call ? scale : __builtin_nanf("");
            last_tile(sc);
        }
    } else {
        waitv<0>();
#pragma unroll
        for (int mi = 0; mi < 4; ++mi) store_rows(mi, __builtin_nanf(""));
    }
}
}

extern "C" void kernel_launch(int rank, int nranks, ncclComm_t comm, hipStream_t stream,
                              void* const* d_in, const int* in_sizes, int n_in,
                              void* d_out, int out_size, const int* in_dtypes, int out_dtype,
                              void* const* d_out_all, void* const* d_ws_all, size_t ws_bytes) {
    using namespace agg;
    (void)comm; (void)in_dtypes; (void)out_dtype; (void)d_out_all;
    if (nranks != NR || n_in != 4 || (unsigned)rank >= (unsigned)NR || ws_bytes < WS_NEED ||
        in_sizes[0] != MSH * KD || in_sizes[1] != KD * NL || out_size != NR * MSH * NL) {
        fprintf(stderr, "ag_gemm: unexpected problem shape\n");
        return;
    }
    hipError_t e = hipFuncSetAttribute((const void*)ag_gemm, hipFuncAttributeMaxDynamicSharedMemorySize, LDS_BYTES);
    if (e != hipSuccess) { fprintf(stderr, "ag_gemm: hipFuncSetAttribute failed: %s\n", hipGetErrorString(e)); return; }
    Params p;
    p.x = (const unsigned char*)d_in[0]; p.w = (const unsigned char*)d_in[1];
    p.sx = (const float*)d_in[2]; p.sw = (const float*)d_in[3]; p.out = (float*)d_out;
    for (int i = 0; i < NR; ++i) p.ws[i] = (char*)d_ws_all[(rank + i) & 7];
    p.rank = rank;
    static const unsigned PERM[NR] = {0x51473602u, 0x12570364u, 0x72536104u, 0x26413705u, 0x74261053u, 0x53164702u, 0x76135240u, 0x14370625u};
    p.perm = PERM[rank];
    hipLaunchKernelGGL(ag_gemm, dim3(256), dim3(512), LDS_BYTES, stream, p);
    e = hipGetLastError();
    if (e != hipSuccess) fprintf(stderr, "ag_gemm: launch failed: %s\n", hipGetErrorString(e));
}
```
